```python
import numpy as np

kernel_name = "gpu_gemm_a2a_m4096_k4096_n16384_f32_relu"


ABSMAX_RTOL = 0.001

COMPUTE_DTYPE = "float32"
EPILOGUE = "relu"
IN_QUANT = None
OUT_QUANT = None
IN_DTYPES = {"x": "float32", "w_mat": "float32"}
WORLD_SIZE = 8

def setup_inputs():

    x = np.random.randn(4096, 4096).astype(np.float32)
    w_mat = (np.random.randn(4096, 16384) / np.sqrt(4096)).astype(np.float32)
    return {"x": x, "w_mat": w_mat}

def _epilogue(y):
    return np.maximum(y, 0.0).astype(np.float32)

def reference(x, w_mat):
    y = (x @ w_mat).astype(np.float32)
    return _epilogue(y)
```

```cpp
#include <hip/hip_runtime.h>
#include <rccl/rccl.h>
#include <cstdio>

#define HIP_CHECK(expr) do { hipError_t _e = (expr); if (_e != hipSuccess) { \
    fprintf(stderr, "HIP error %s at %s:%d\n", hipGetErrorString(_e), __FILE__, __LINE__); \
    return; } } while (0)

namespace {

typedef _Float16 f16x2 __attribute__((ext_vector_type(2)));
typedef _Float16 f16x8 __attribute__((ext_vector_type(8)));
typedef float f32x2 __attribute__((ext_vector_type(2)));
typedef float f32x4 __attribute__((ext_vector_type(4)));
typedef unsigned u32x4 __attribute__((ext_vector_type(4)));
#define LAS __attribute__((address_space(3)))
#define GAS __attribute__((address_space(1)))

constexpr int NR = 8;
constexpr int MS = 512;
constexpr int KD = 4096;
constexpr int NT = 16384;
constexpr int NS = 2048;
constexpr int BM = 256, BN = 128, BK = 64;
constexpr int NKT = KD / BK;
constexpr int A_TILE = BM * BK * 2;
constexpr int B_TILE = BN * BK * 2;
constexpr int NABUF = 3;
constexpr int LDS_A = 0;
constexpr int LDS_B = NABUF * A_TILE;
constexpr int LDS_MISC = LDS_B + 2 * B_TILE;
constexpr int LDS_BYTES = LDS_MISC + 64;
constexpr int GRID = 256;
constexpr int NCONS = 512;
constexpr int NTHREADS = 768;

constexpr size_t SLOT_BYTES = (size_t)MS * KD * 2;
constexpr size_t WS_X16 = 0;
constexpr size_t WS_FLAGS = 2 * SLOT_BYTES;
constexpr size_t WS_CNT = WS_FLAGS + (size_t)NR * GRID * 4;
constexpr size_t WS_TMO = WS_CNT + (size_t)GRID * 128;
constexpr size_t WS_NEED = WS_TMO + 128;

constexpr unsigned long long DEADLINE_TICKS = 30000000ull;

struct Params {
    const float* x;
    const float* w;
    float* out;
    char* ws[NR];
    int rank;
    int pad;
};

__device__ __forceinline__ f16x8 cvt8(f32x4 a, f32x4 b) {
    f16x2 p0 = __builtin_convertvector((f32x2){a[0], a[1]}, f16x2);
    f16x2 p1 = __builtin_convertvector((f32x2){a[2], a[3]}, f16x2);
    f16x2 p2 = __builtin_convertvector((f32x2){b[0], b[1]}, f16x2);
    f16x2 p3 = __builtin_convertvector((f32x2){b[2], b[3]}, f16x2);
    return (f16x8){p0[0], p0[1], p1[0], p1[1], p2[0], p2[1], p3[0], p3[1]};
}

#define WAIT_V(n) asm volatile("s_waitcnt vmcnt(" #n ")" ::: "memory")
#define WAIT_L(n) asm volatile("s_waitcnt lgkmcnt(" #n ")" ::: "memory")
#define BAR() do { asm volatile("" ::: "memory"); __builtin_amdgcn_s_barrier(); asm volatile("" ::: "memory"); } while (0)

__global__ void __launch_bounds__(NTHREADS, 3) gemm_pull_kernel(Params p) {
    extern __shared__ __attribute__((aligned(16))) char smem[];
    LAS char* lds = (LAS char*)smem;
    const int tid = threadIdx.x;
    const int wid = __builtin_amdgcn_readfirstlane(tid >> 6);
    const int lane = tid & 63;
    const int b = blockIdx.x;
    const int label = b & 7, rr = b >> 3;
    const int q = rr >> 4, ct = rr & 15;
    const unsigned src = (unsigned)(p.rank + label) & 7u;
    char* const myws = p.ws[p.rank];
    GAS unsigned* const cnt = (GAS unsigned*)(myws + WS_CNT + (size_t)b * 128);
    GAS unsigned* const tmo = (GAS unsigned*)(myws + WS_TMO);
    LAS unsigned* const misc = (LAS unsigned*)(lds + LDS_MISC);

    if (wid < 8) {
        unsigned call;
        size_t slot;
        {
            const int pq = b >> 7, pt = (b >> 1) & 63, ph = b & 1;
            f32x4 v[2][2];
#pragma unroll
            for (int i = 0; i < 2; ++i) {
                const int idx = tid + i * NCONS, row = ph * 128 + (idx >> 3), ch = idx & 7;
                const float* g = p.x + (size_t)(pq * 256 + row) * KD + pt * BK + ch * 8;
                v[i][0] = *(const f32x4*)g;
                v[i][1] = *(const f32x4*)(g + 4);
            }
            asm volatile("" ::: "memory");
            call = __builtin_amdgcn_readfirstlane(__hip_atomic_load(cnt, __ATOMIC_RELAXED, __HIP_MEMORY_SCOPE_AGENT)) + 1u;
            slot = WS_X16 + (size_t)(call & 1u) * SLOT_BYTES;
            char* dst = myws + slot + (size_t)(pq * NKT + pt) * A_TILE;
#pragma unroll
            for (int i = 0; i < 2; ++i) {
                const int idx = tid + i * NCONS, row = ph * 128 + (idx >> 3), ch = idx & 7;
                const f16x8 h = cvt8(v[i][0], v[i][1]);
                char* d = dst + row * 128 + ((ch ^ ((row >> 1) & 7)) << 4);
                asm volatile("global_store_dwordx4 %0, %1, off sc0 sc1\n\ts_nop 1"
                             :: "v"(d), "v"(__builtin_bit_cast(u32x4, h)) : "memory");
            }
        }
        WAIT_V(0);
        BAR();
        if (wid == 0) {
            if (lane < NR) {
                GAS unsigned* f = (GAS unsigned*)(p.ws[lane] + WS_FLAGS) + p.rank * GRID + b;
                __hip_atomic_store(f, call, __ATOMIC_RELAXED, __HIP_MEMORY_SCOPE_SYSTEM);
            }
            GAS unsigned long long* fl = (GAS unsigned long long*)(myws + WS_FLAGS + ((size_t)src * GRID + q * 128) * 4) + lane;
            const unsigned long long t0 = __builtin_amdgcn_s_memrealtime();
            for (;;) {
                const unsigned long long f = __hip_atomic_load(fl, __ATOMIC_RELAXED, __HIP_MEMORY_SCOPE_SYSTEM);
                const bool ok = (unsigned)f >= call && (unsigned)(f >> 32) >= call;
                if (__all(ok)) break;
                if (__hip_atomic_load(tmo, __ATOMIC_RELAXED, __HIP_MEMORY_SCOPE_AGENT) == call ||
                    __builtin_amdgcn_s_memrealtime() - t0 > DEADLINE_TICKS) {
                    __hip_atomic_store(tmo, call, __ATOMIC_RELAXED, __HIP_MEMORY_SCOPE_AGENT);
                    break;
                }
                __builtin_amdgcn_s_sleep(2);
            }
            __builtin_amdgcn_fence(__ATOMIC_ACQUIRE, "");
            WAIT_V(0);
            const unsigned bad = __hip_atomic_load(tmo, __ATOMIC_RELAXED, __HIP_MEMORY_SCOPE_AGENT) == call ? 1u : 0u;
            if (lane == 0) misc[0] = bad;
            WAIT_L(0);
        }
        BAR();
        const bool bad = misc[0] != 0u;

        const int wm = wid >> 1, wn = wid & 1;
        const int fr = lane & 15, fq = lane >> 4;
        f32x4 acc[4][4];
#pragma unroll
        for (int i = 0; i < 4; ++i)
#pragma unroll
            for (int j = 0; j < 4; ++j) acc[i][j] = (f32x4){0.f, 0.f, 0.f, 0.f};

        if (!bad) {
            const char* asrc = p.ws[src] + slot + (size_t)q * NKT * A_TILE + tid * 16;
            auto stageA = [&](int t, int buf) {
                const char* g = asrc + (size_t)t * A_TILE;
                LAS char* l = lds + LDS_A + buf * A_TILE + wid * 1024;
#pragma unroll
                for (int i = 0; i < 4; ++i)
                    __builtin_amdgcn_global_load_lds((const GAS unsigned*)(g + i * 8192), (LAS unsigned*)(l + i * 8192), 16, 0, 0);
            };
            const int sw = (fr >> 1) & 7;
            const int aoff0 = (wm * 64 + fr) * 128 + ((fq ^ sw) << 4);
            const int boff0 = LDS_B + (wn * 64 + fr) * 128 + ((fq ^ sw) << 4);
            stageA(0, 0);
            stageA(1, 1);
            int abuf = 0;
            for (int t = 0; t < NKT; ++t) {
                if (t + 1 < NKT) WAIT_V(4); else WAIT_V(0);
                WAIT_L(0);
                BAR();
                int nb = abuf + 2; if (nb >= NABUF) nb -= NABUF;
                if (t + 2 < NKT) stageA(t + 2, nb);
                const LAS char* A = lds + LDS_A + abuf * A_TILE;
                const LAS char* B = lds + (t & 1) * B_TILE;
#pragma unroll
                for (int ks = 0; ks < 2; ++ks) {
                    f16x8 af[4], bf[4];
#pragma unroll
                    for (int i = 0; i < 4; ++i) bf[i] = *(const LAS f16x8*)(B + ((boff0 + i * 2048) ^ (ks * 64)));
#pragma unroll
                    for (int i = 0; i < 4; ++i) af[i] = *(const LAS f16x8*)(A + ((aoff0 + i * 2048) ^ (ks * 64)));
#pragma unroll
                    for (int i = 0; i < 4; ++i)
#pragma unroll
                        for (int j = 0; j < 4; ++j)
                            acc[i][j] = __builtin_amdgcn_mfma_f32_16x16x32_f16(bf[j], af[i], acc[i][j], 0, 0, 0);
                }
                abuf = abuf + 1; if (abuf >= NABUF) abuf = 0;
            }
        }
        const float qnan = __builtin_nanf("");
        float* o = p.out + (size_t)(src * MS + q * BM + wm * 64 + fr) * NS + ct * BN + wn * 64 + fq * 4;
#pragma unroll
        for (int i = 0; i < 4; ++i)
#pragma unroll
            for (int j = 0; j < 4; ++j) {
                f32x4 v = acc[i][j];
                v[0] = fmaxf(v[0], 0.f); v[1] = fmaxf(v[1], 0.f); v[2] = fmaxf(v[2], 0.f); v[3] = fmaxf(v[3], 0.f);
                if (bad) v = (f32x4){qnan, qnan, qnan, qnan};
                *(f32x4*)(o + (size_t)i * 16 * NS + j * 16) = v;
            }
        if (tid == 0) __hip_atomic_store(cnt, call, __ATOMIC_RELAXED, __HIP_MEMORY_SCOPE_AGENT);
    } else {
        const int pt = tid - NCONS;
        const int c = pt & 31, kg = pt >> 5;
        const float* wp = p.w + (size_t)(kg * 8) * NT + p.rank * NS + ct * BN + c * 4;
        f32x4 wa[8], wb[8];
        auto loadW = [&](f32x4 (&w)[8], int t) {
            const float* g = wp + (size_t)t * BK * NT;
#pragma unroll
            for (int e = 0; e < 8; ++e) w[e] = *(const f32x4*)(g + (size_t)e * NT);
        };
        auto storeB = [&](const f32x4 (&w)[8], int buf) {
            LAS char* B = lds + LDS_B + buf * B_TILE;
#pragma unroll
            for (int i = 0; i < 4; ++i) {
                const int row = 4 * c + i;
                const f16x8 h = cvt8((f32x4){w[0][i], w[1][i], w[2][i], w[3][i]}, (f32x4){w[4][i], w[5][i], w[6][i], w[7][i]});
                *(LAS f16x8*)(B + row * 128 + ((kg ^ ((row >> 1) & 7)) << 4)) = h;
            }
        };
        loadW(wa, 0);
        loadW(wb, 1);
        BAR();
        storeB(wa, 0);
        loadW(wa, 2);
        WAIT_L(0);
        BAR();
        const bool bad = misc[0] != 0u;
        if (!bad) {
            for (int t = 0; t < NKT; t += 2) {
                BAR();
                storeB(wb, 1);
                loadW(wb, t + 3 < NKT ? t + 3 : NKT - 1);
                WAIT_L(0);
                BAR();
                if (t + 2 < NKT) storeB(wa, 0);
                loadW(wa, t + 4 < NKT ? t + 4 : NKT - 1);
                WAIT_L(0);
            }
        }
    }
}

}

extern "C" void kernel_launch(int rank, int nranks, ncclComm_t comm,
                              hipStream_t stream,
                              void* const* d_in, const int* in_sizes, int n_in,
                              void* d_out, int out_size,
                              const int* in_dtypes, int out_dtype,
                              void* const* d_out_all,
                              void* const* d_ws_all, size_t ws_bytes) {
    (void)comm; (void)in_dtypes; (void)out_dtype; (void)d_out_all;
    if (nranks != NR || n_in < 2 || (unsigned)rank >= (unsigned)NR || ws_bytes < WS_NEED ||
        in_sizes[0] != MS * KD || in_sizes[1] != KD * NT || out_size != NR * MS * NS) {
        fprintf(stderr, "kernel_launch: unexpected shapes\n");
        return;
    }
    static bool attr_set[64] = {};
    int dev = 0;
    HIP_CHECK(hipGetDevice(&dev));
    if (dev >= 0 && dev < 64 && !attr_set[dev]) {
        HIP_CHECK(hipFuncSetAttribute((const void*)gemm_pull_kernel, hipFuncAttributeMaxDynamicSharedMemorySize, LDS_BYTES));
        attr_set[dev] = true;
    }
    Params p;
    p.x = (const float*)d_in[0];
    p.w = (const float*)d_in[1];
    p.out = (float*)d_out;
    for (int i = 0; i < NR; ++i) p.ws[i] = (char*)d_ws_all[i];
    p.rank = rank;
    p.pad = 0;
    gemm_pull_kernel<<<GRID, NTHREADS, LDS_BYTES, stream>>>(p);
    HIP_CHECK(hipGetLastError());
}
```

```cpp
#include <hip/hip_runtime.h>
#include <rccl/rccl.h>
#include <cstdio>

#define HIP_CHECK(expr) do { hipError_t _e = (expr); if (_e != hipSuccess) { \
    fprintf(stderr, "HIP error %s at %s:%d\n", hipGetErrorString(_e), __FILE__, __LINE__); \
    return; } } while (0)

namespace {

typedef _Float16 f16x2 __attribute__((ext_vector_type(2)));
typedef _Float16 f16x8 __attribute__((ext_vector_type(8)));
typedef float f32x2 __attribute__((ext_vector_type(2)));
typedef float f32x4 __attribute__((ext_vector_type(4)));
typedef unsigned u32x4 __attribute__((ext_vector_type(4)));
#define LAS __attribute__((address_space(3)))
#define GAS __attribute__((address_space(1)))

constexpr int NR = 8;
constexpr int MS = 512;
constexpr int KD = 4096;
constexpr int NT = 16384;
constexpr int NS = 2048;
constexpr int BM = 256, BN = 144, BK = 64;
constexpr int NCT = 15;
constexpr int NKT = KD / BK;
constexpr int A_TILE = BM * BK * 2;
constexpr int B_TILE = BN * BK * 2;
constexpr int NBUF = 3;
constexpr int LDS_A = 0;
constexpr int LDS_B = NBUF * A_TILE;
constexpr int LDS_MISC = LDS_B + NBUF * B_TILE;
constexpr int LDS_BYTES = LDS_MISC + 64;
constexpr int GRID = 256;
constexpr int NTHREADS = 512;

constexpr size_t SLOT_BYTES = (size_t)MS * KD * 2;
constexpr size_t WS_X16 = 0;
constexpr size_t WS_FLAGS = 2 * SLOT_BYTES;
constexpr size_t WS_WFLAGS = WS_FLAGS + (size_t)NR * GRID * 4;
constexpr size_t WS_CNT = WS_WFLAGS + 16 * 64 * 4;
constexpr size_t WS_TMO = WS_CNT + (size_t)GRID * 128;
constexpr size_t WS_W16 = 9u << 20;
constexpr size_t WS_NEED = WS_W16 + (size_t)NCT * NKT * B_TILE;
static_assert(WS_TMO + 128 <= WS_W16, "workspace layout");

constexpr unsigned long long DEADLINE_TICKS = 30000000ull;

struct Params {
    const float* x;
    const float* w;
    float* out;
    char* ws[NR];
    int rank;
    int pad;
};

template <int V> struct IC { static constexpr int value = V; };

__device__ __forceinline__ f16x8 cvt8(f32x4 a, f32x4 b) {
    f16x2 p0 = __builtin_convertvector((f32x2){a[0], a[1]}, f16x2);
    f16x2 p1 = __builtin_convertvector((f32x2){a[2], a[3]}, f16x2);
    f16x2 p2 = __builtin_convertvector((f32x2){b[0], b[1]}, f16x2);
    f16x2 p3 = __builtin_convertvector((f32x2){b[2], b[3]}, f16x2);
    return (f16x8){p0[0], p0[1], p1[0], p1[1], p2[0], p2[1], p3[0], p3[1]};
}

__device__ __forceinline__ void glds16(unsigned voff, const char* sbase, unsigned ldsaddr) {
    asm volatile("s_mov_b32 m0, %2\n\ts_nop 0\n\tglobal_load_lds_dwordx4 %0, %1" :: "v"(voff), "s"(sbase), "s"(ldsaddr) : "memory");
}
__device__ __forceinline__ void glds16_sc(unsigned voff, const char* sbase, unsigned ldsaddr) {
    asm volatile("s_mov_b32 m0, %2\n\ts_nop 0\n\tglobal_load_lds_dwordx4 %0, %1 sc0 sc1" :: "v"(voff), "s"(sbase), "s"(ldsaddr) : "memory");
}
__device__ __forceinline__ void store_wt(void* d, f16x8 h) {
    asm volatile("global_store_dwordx4 %0, %1, off sc0 sc1\n\ts_nop 1" :: "v"(d), "v"(__builtin_bit_cast(u32x4, h)) : "memory");
}

#define WAIT_V(n) asm volatile("s_waitcnt vmcnt(" #n ")" ::: "memory")
#define WAIT_L(n) asm volatile("s_waitcnt lgkmcnt(" #n ")" ::: "memory")
#define BAR() do { asm volatile("" ::: "memory"); __builtin_amdgcn_s_barrier(); asm volatile("" ::: "memory"); } while (0)
#define SCHED() __builtin_amdgcn_sched_barrier(0)

#define STAMP(i) do {} while (0)
#define LSTAMP_DECL do {} while (0)
#define LSTAMP(t) do {} while (0)
#define LSTAMP_FLUSH() do {} while (0)

__global__ void __launch_bounds__(NTHREADS, 2) gemm_pull_kernel(Params p) {
    extern __shared__ __attribute__((aligned(16))) char smem[];
    LAS char* lds = (LAS char*)smem;
    const unsigned lds0 = (unsigned)(size_t)lds;
    const int tid = threadIdx.x;
    const int wid = __builtin_amdgcn_readfirstlane(tid >> 6);
    const int lane = tid & 63;
    const int b = blockIdx.x;
    const int label = b & 7, rr = b >> 3;
    const bool puller = rr >= 2 * NCT;
    const int q = puller ? rr - 2 * NCT : rr / NCT;
    const int ct = puller ? 0 : rr - q * NCT;
    const unsigned src = (unsigned)(p.rank + label) & 7u;
    char* const myws = p.ws[p.rank];
    GAS unsigned* const cnt = (GAS unsigned*)(myws + WS_CNT + (size_t)b * 128);
    GAS unsigned* const tmo = (GAS unsigned*)(myws + WS_TMO);
    LAS unsigned* const misc = (LAS unsigned*)(lds + LDS_MISC);

    STAMP(0);
    unsigned call;
    size_t slot;
    GAS unsigned* fdst = nullptr;
    {
        const int pq = b >> 7, pt = (b >> 1) & 63, ph = b & 1;
        f32x4 v[2][2];
#pragma unroll
        for (int i = 0; i < 2; ++i) {
            const int idx = tid + i * NTHREADS, row = ph * 128 + (idx >> 3), ch = idx & 7;
            const float* g = p.x + (size_t)(pq * 256 + row) * KD + pt * BK + ch * 8;
            v[i][0] = *(const f32x4*)g;
            v[i][1] = *(const f32x4*)(g + 4);
        }
        if (wid == 0) fdst = (GAS unsigned*)(p.ws[lane & 7] + WS_FLAGS) + p.rank * GRID + b;
        const unsigned prev = __hip_atomic_load(cnt, __ATOMIC_RELAXED, __HIP_MEMORY_SCOPE_AGENT);
        if (wid == 0) asm volatile("buffer_inv sc0 sc1" ::: "memory");
        asm volatile("" ::: "memory");
        call = __builtin_amdgcn_readfirstlane(prev) + 1u;
        slot = WS_X16 + (size_t)(call & 1u) * SLOT_BYTES;
        char* dst = myws + slot + (size_t)(pq * NKT + pt) * A_TILE;
#pragma unroll
        for (int i = 0; i < 2; ++i) {
            const int idx = tid + i * NTHREADS, row = ph * 128 + (idx >> 3), ch = idx & 7;
            store_wt(dst + row * 128 + ((ch ^ ((row >> 1) & 7)) << 4), cvt8(v[i][0], v[i][1]));
        }
    }
    WAIT_V(0);
    STAMP(1);
    BAR();
    if (wid == 0) {
        if (lane < NR) __hip_atomic_store(fdst, call, __ATOMIC_RELAXED, __HIP_MEMORY_SCOPE_SYSTEM);
    }
    char* const w16 = myws + WS_W16 + (size_t)ct * NKT * B_TILE;
    if (!puller) {
        const int kt0 = label * 2 + q;
        const int colbase = p.rank * NS + ct * BN;
        const int colmax = p.rank * NS + NS - 4;
        f32x4 w[3][8];
#pragma unroll
        for (int j = 0; j < 3; ++j) {
            const int I = tid + j * NTHREADS;
            if (I < 4 * 288) {
                const int r = I / 288, rem = I - r * 288, kg = rem & 7, c = rem >> 3;
                const float* g = p.w + (size_t)((kt0 + 16 * r) * BK + kg * 8) * NT + min(colbase + c * 4, colmax);
#pragma unroll
                for (int e = 0; e < 8; ++e) w[j][e] = __builtin_nontemporal_load((const f32x4*)(g + (size_t)e * NT));
            }
        }
#pragma unroll
        for (int j = 0; j < 3; ++j) {
            const int I = tid + j * NTHREADS;
            if (I < 4 * 288) {
                const int r = I / 288, rem = I - r * 288, kg = rem & 7, c = rem >> 3;
                char* B = w16 + (size_t)(kt0 + 16 * r) * B_TILE;
#pragma unroll
                for (int i = 0; i < 4; ++i) {
                    const int row = 4 * c + i;
                    store_wt(B + row * 128 + ((kg ^ ((row >> 1) & 7)) << 4),
                             cvt8((f32x4){w[j][0][i], w[j][1][i], w[j][2][i], w[j][3][i]}, (f32x4){w[j][4][i], w[j][5][i], w[j][6][i], w[j][7][i]}));
                }
            }
        }
        WAIT_V(0);
        STAMP(7);
        BAR();
    }
    if (wid == 0) {
        GAS unsigned* wf = (GAS unsigned*)(myws + WS_WFLAGS) + ct * 64;
        if (!puller && lane < 4) __hip_atomic_store(wf + label * 2 + q + 16 * lane, call, __ATOMIC_RELAXED, __HIP_MEMORY_SCOPE_SYSTEM);
        GAS unsigned long long* fl = (GAS unsigned long long*)(myws + WS_FLAGS + ((size_t)src * GRID + q * 128) * 4) + lane;
        const unsigned long long t0 = __builtin_amdgcn_s_memrealtime();
        for (;;) {
            const unsigned long long f = __hip_atomic_load(fl, __ATOMIC_RELAXED, __HIP_MEMORY_SCOPE_SYSTEM);
            const unsigned g = puller ? call : __hip_atomic_load(wf + lane, __ATOMIC_RELAXED, __HIP_MEMORY_SCOPE_SYSTEM);
            const bool ok = (unsigned)f >= call && (unsigned)(f >> 32) >= call && g >= call;
            if (__all(ok)) break;
            if (__hip_atomic_load(tmo, __ATOMIC_RELAXED, __HIP_MEMORY_SCOPE_AGENT) == call ||
                __builtin_amdgcn_s_memrealtime() - t0 > DEADLINE_TICKS) {
                __hip_atomic_store(tmo, call, __ATOMIC_RELAXED, __HIP_MEMORY_SCOPE_AGENT);
                break;
            }
            __builtin_amdgcn_s_sleep(1);
        }
        STAMP(2);
        WAIT_V(0);
        const unsigned bad = __hip_atomic_load(tmo, __ATOMIC_RELAXED, __HIP_MEMORY_SCOPE_AGENT) == call ? 1u : 0u;
        if (lane == 0) misc[0] = bad;
        WAIT_L(0);
        STAMP(3);
    }
    BAR();
    const bool bad = misc[0] != 0u;
    const char* asrc = p.ws[src] + slot + (size_t)q * NKT * A_TILE;
    unsigned voff[4];
#pragma unroll
    for (int i = 0; i < 4; ++i) voff[i] = (unsigned)tid * 16u + i * 8192u;

    if (puller) {
        if (!bad) {
            const unsigned l = lds0 + LDS_A + wid * 1024;
            LSTAMP_DECL;
            for (int t = 0; t < NKT; ++t) {
                const char* g = asrc + (size_t)t * A_TILE;
#pragma unroll
                for (int i = 0; i < 4; ++i) glds16(voff[i], g, l + i * 8192);
                WAIT_V(4);
                BAR();
                LSTAMP(t);
            }
            WAIT_V(0);
            LSTAMP_FLUSH();
        }
        STAMP(4);
    } else {
        const int fr = lane & 15, fq = lane >> 4;
        f32x4 acc[2][9];
#pragma unroll
        for (int i = 0; i < 2; ++i)
#pragma unroll
            for (int j = 0; j < 9; ++j) acc[i][j] = (f32x4){0.f, 0.f, 0.f, 0.f};

        if (!bad) {
            const int sw = (fr >> 1) & 7;
            const int aoff0 = LDS_A + (wid * 32 + fr) * 128 + ((fq ^ sw) << 4), aoff1 = aoff0 ^ 64;
            const int boff0 = LDS_B + fr * 128 + ((fq ^ sw) << 4), boff1 = boff0 ^ 64;
            const unsigned ldsa = lds0 + LDS_A + wid * 1024, ldsb = lds0 + LDS_B + wid * 1024;
            const bool third = wid < 2;
            f16x8 a0[2], a1[2], b0[3], b1[3];
#define LDA(dst, aoff, ab) do { _Pragma("unroll") for (int i_ = 0; i_ < 2; ++i_) dst[i_] = *(const LAS f16x8*)(lds + (ab) + (aoff) + i_ * 2048); } while (0)
#define LDB(dst, boff, bb, g) do { _Pragma("unroll") for (int j_ = 0; j_ < 3; ++j_) dst[j_] = *(const LAS f16x8*)(lds + (bb) + (boff) + ((g) * 3 + j_) * 2048); } while (0)
#define MMA(g, bf, af) do { _Pragma("unroll") for (int j_ = 0; j_ < 3; ++j_) _Pragma("unroll") for (int i_ = 0; i_ < 2; ++i_) \
        acc[i_][(g) * 3 + j_] = __builtin_amdgcn_mfma_f32_16x16x32_f16(bf[j_], af[i_], acc[i_][(g) * 3 + j_], 0, 0, 0); } while (0)
#pragma unroll
            for (int t = 0; t < 2; ++t) {
                if (third) glds16_sc(voff[2], w16 + (size_t)t * B_TILE, ldsb + t * B_TILE + 16384);
                glds16_sc(voff[0], w16 + (size_t)t * B_TILE, ldsb + t * B_TILE);
                glds16_sc(voff[1], w16 + (size_t)t * B_TILE, ldsb + t * B_TILE + 8192);
#pragma unroll
                for (int i = 0; i < 4; ++i) glds16(voff[i], asrc + (size_t)t * A_TILE, ldsa + t * A_TILE + i * 8192);
            }
            if (third) glds16_sc(voff[2], w16 + (size_t)2 * B_TILE, ldsb + 2 * B_TILE + 16384);
            glds16_sc(voff[0], w16 + (size_t)2 * B_TILE, ldsb + 2 * B_TILE);
            if (third) WAIT_V(9); else WAIT_V(7);
            BAR();
            LDA(a0, aoff0, 0);
            LDB(b0, boff0, 0, 0);
            LSTAMP_DECL;
            int buf = 0;
            auto iter = [&](auto mode_, int t) {
                constexpr int MODE = decltype(mode_)::value;
                const int bn = (buf == 2) ? 0 : buf + 1;
                const int bp = (buf == 0) ? 2 : buf - 1;
                const int ab = buf * A_TILE, bb = buf * B_TILE;
                const char* ga = asrc + (size_t)(t + 2) * A_TILE;
                const char* gb = w16 + (size_t)(t + 2) * B_TILE;
                const unsigned la = ldsa + bp * A_TILE, lb = ldsb + bp * B_TILE;
                SCHED();
                LDB(b1, boff0, bb, 1); LDA(a1, aoff1, ab);
                SCHED();
                MMA(0, b0, a0);
                if (MODE <= 1) glds16_sc(voff[1], gb, lb + 8192);
                SCHED();
                LDB(b0, boff0, bb, 2);
                SCHED();
                MMA(1, b1, a0);
                if (MODE <= 1) glds16(voff[0], ga, la);
                SCHED();
                LDB(b1, boff1, bb, 0);
                SCHED();
                MMA(2, b0, a0);
                if (MODE <= 1) glds16(voff[1], ga, la + 8192);
                SCHED();
                LDB(b0, boff1, bb, 1);
                SCHED();
                MMA(0, b1, a1);
                if (MODE <= 1) glds16(voff[2], ga, la + 2 * 8192);
                SCHED();
                LDB(b1, boff1, bb, 2);
                SCHED();
                MMA(1, b0, a1);
                if (MODE <= 1) glds16(voff[3], ga, la + 3 * 8192);
                SCHED();
                if (MODE <= 2) {
                    if (MODE <= 1) { if (third) WAIT_V(7); else WAIT_V(6); } else WAIT_V(0);
                    WAIT_L(0);
                    LSTAMP(t);
                    BAR();
                    LDA(a0, aoff0, bn * A_TILE);
                    LDB(b0, boff0, bn * B_TILE, 0);
                    SCHED();
                }
                MMA(2, b1, a1);
                if (MODE == 0) {
                    if (third) glds16_sc(voff[2], gb + B_TILE, ldsb + bb + 16384);
                    glds16_sc(voff[0], gb + B_TILE, ldsb + bb);
                }
                buf = bn;
            };
            for (int t = 0; t < NKT - 3; ++t) iter(IC<0>{}, t);
            iter(IC<1>{}, NKT - 3);
            iter(IC<2>{}, NKT - 2);
            iter(IC<3>{}, NKT - 1);
            LSTAMP_FLUSH();
        }
        STAMP(4);
        const float qnan = __builtin_nanf("");
        float* o = p.out + (size_t)(src * MS + q * BM + wid * 32 + fr) * NS + ct * BN + fq * 4;
        const int nvalid = (NS - ct * BN) >> 4;
#pragma unroll
        for (int i = 0; i < 2; ++i)
#pragma unroll
            for (int j = 0; j < 9; ++j) {
                f32x4 v = acc[i][j];
                v[0] = fmaxf(v[0], 0.f); v[1] = fmaxf(v[1], 0.f); v[2] = fmaxf(v[2], 0.f); v[3] = fmaxf(v[3], 0.f);
                if (bad) v = (f32x4){qnan, qnan, qnan, qnan};
                if (j < nvalid) *(f32x4*)(o + (size_t)i * 16 * NS + j * 16) = v;
            }
        STAMP(5);
    }
    if (tid == 0) __hip_atomic_store(cnt, call, __ATOMIC_RELAXED, __HIP_MEMORY_SCOPE_AGENT);
}

}

extern "C" void kernel_launch(int rank, int nranks, ncclComm_t comm,
                              hipStream_t stream,
                              void* const* d_in, const int* in_sizes, int n_in,
                              void* d_out, int out_size,
                              const int* in_dtypes, int out_dtype,
                              void* const* d_out_all,
                              void* const* d_ws_all, size_t ws_bytes) {
    (void)comm; (void)in_dtypes; (void)out_dtype; (void)d_out_all;
    if (nranks != NR || n_in < 2 || (unsigned)rank >= (unsigned)NR || ws_bytes < WS_NEED ||
        in_sizes[0] != MS * KD || in_sizes[1] != KD * NT || out_size != NR * MS * NS) {
        fprintf(stderr, "kernel_launch: unexpected shapes\n");
        return;
    }
    static bool attr_set[64] = {};
    int dev = 0;
    HIP_CHECK(hipGetDevice(&dev));
    if (dev >= 0 && dev < 64 && !attr_set[dev]) {
        HIP_CHECK(hipFuncSetAttribute((const void*)gemm_pull_kernel, hipFuncAttributeMaxDynamicSharedMemorySize, LDS_BYTES));
        attr_set[dev] = true;
    }
    Params p;
    p.x = (const float*)d_in[0];
    p.w = (const float*)d_in[1];
    p.out = (float*)d_out;
    for (int i = 0; i < NR; ++i) p.ws[i] = (char*)d_ws_all[i];
    p.rank = rank;
    p.pad = 0;
    gemm_pull_kernel<<<GRID, NTHREADS, LDS_BYTES, stream>>>(p);
    HIP_CHECK(hipGetLastError());
}
```

```cpp
#include <hip/hip_runtime.h>
#include <rccl/rccl.h>
#include <cstdio>

#define HIP_CHECK(expr) do { hipError_t _e = (expr); if (_e != hipSuccess) { \
    fprintf(stderr, "HIP error %s at %s:%d\n", hipGetErrorString(_e), __FILE__, __LINE__); \
    return; } } while (0)

namespace {

typedef _Float16 f16x2 __attribute__((ext_vector_type(2)));
typedef _Float16 f16x8 __attribute__((ext_vector_type(8)));
typedef float f32x2 __attribute__((ext_vector_type(2)));
typedef float f32x4 __attribute__((ext_vector_type(4)));
typedef unsigned u32x4 __attribute__((ext_vector_type(4)));
#define LAS __attribute__((address_space(3)))
#define GAS __attribute__((address_space(1)))

constexpr int NR = 8;
constexpr int MS = 512;
constexpr int KD = 4096;
constexpr int NT = 16384;
constexpr int NS = 2048;
constexpr int BM = 256, BN = 144, BK = 64;
constexpr int NCT = 15;
constexpr int NKT = KD / BK;
constexpr int A_TILE = BM * BK * 2;
constexpr int B_TILE = BN * BK * 2;
constexpr int NBUF = 3;
constexpr int LDS_A = 0;
constexpr int LDS_B = NBUF * A_TILE;
constexpr int LDS_MISC = LDS_B + NBUF * B_TILE;
constexpr int LDS_BYTES = LDS_MISC + 64;
constexpr int GRID = 256;
constexpr int NCONS = 512;
constexpr int NTHREADS = 768;

constexpr size_t SLOT_BYTES = (size_t)MS * KD * 2;
constexpr size_t WS_X16 = 0;
constexpr size_t WS_FLAGS = 2 * SLOT_BYTES;
constexpr size_t WS_WFLAGS = WS_FLAGS + (size_t)NR * GRID * 4;
constexpr size_t WS_CNT = WS_WFLAGS + 16 * 64 * 4 * 4;
constexpr size_t WS_TMO = WS_CNT + (size_t)GRID * 128;
constexpr size_t WS_W16 = 9u << 20;
constexpr size_t WS_NEED = WS_W16 + (size_t)NCT * NKT * B_TILE;
static_assert(WS_TMO + 128 <= WS_W16, "workspace layout");

constexpr unsigned long long DEADLINE_TICKS = 30000000ull;

struct Params {
    const float* x;
    const float* w;
    float* out;
    char* ws[NR];
    int rank;
    int pad;
};

template <int V> struct IC { static constexpr int value = V; };

__device__ __forceinline__ f16x8 cvt8(f32x4 a, f32x4 b) {
    f16x2 p0 = __builtin_convertvector((f32x2){a[0], a[1]}, f16x2);
    f16x2 p1 = __builtin_convertvector((f32x2){a[2], a[3]}, f16x2);
    f16x2 p2 = __builtin_convertvector((f32x2){b[0], b[1]}, f16x2);
    f16x2 p3 = __builtin_convertvector((f32x2){b[2], b[3]}, f16x2);
    return (f16x8){p0[0], p0[1], p1[0], p1[1], p2[0], p2[1], p3[0], p3[1]};
}

__device__ __forceinline__ void glds16(unsigned voff, const char* sbase, unsigned ldsaddr) {
    asm volatile("s_mov_b32 m0, %2\n\ts_nop 0\n\tglobal_load_lds_dwordx4 %0, %1" :: "v"(voff), "s"(sbase), "s"(ldsaddr) : "memory");
}
__device__ __forceinline__ void glds16_sc(unsigned voff, const char* sbase, unsigned ldsaddr) {
    asm volatile("s_mov_b32 m0, %2\n\ts_nop 0\n\tglobal_load_lds_dwordx4 %0, %1 sc0 sc1" :: "v"(voff), "s"(sbase), "s"(ldsaddr) : "memory");
}
__device__ __forceinline__ void store_wt(void* d, f16x8 h) {
    asm volatile("global_store_dwordx4 %0, %1, off sc0 sc1\n\ts_nop 1" :: "v"(d), "v"(__builtin_bit_cast(u32x4, h)) : "memory");
}

#define WAIT_V(n) asm volatile("s_waitcnt vmcnt(" #n ")" ::: "memory")
#define WAIT_L(n) asm volatile("s_waitcnt lgkmcnt(" #n ")" ::: "memory")
#define BAR() do { asm volatile("" ::: "memory"); __builtin_amdgcn_s_barrier(); asm volatile("" ::: "memory"); } while (0)
#define SCHED() __builtin_amdgcn_sched_barrier(0)

#define STAMP(i) do {} while (0)
#define LSTAMP_DECL do {} while (0)
#define LSTAMP(t) do {} while (0)
#define LSTAMP_FLUSH() do {} while (0)

__global__ void __launch_bounds__(NTHREADS, 3) gemm_pull_kernel(Params p) {
    extern __shared__ __attribute__((aligned(16))) char smem[];
    LAS char* lds = (LAS char*)smem;
    const unsigned lds0 = (unsigned)(size_t)lds;
    const int tid = threadIdx.x;
    const int wid = __builtin_amdgcn_readfirstlane(tid >> 6);
    const int lane = tid & 63;
    const int b = blockIdx.x;
    const int label = b & 7, rr = b >> 3;
    const bool puller = rr >= 2 * NCT;
    const int q = puller ? rr - 2 * NCT : rr / NCT;
    const int ct = puller ? 0 : rr - q * NCT;
    const unsigned src = (unsigned)(p.rank + label) & 7u;
    char* const myws = p.ws[p.rank];
    GAS unsigned* const cnt = (GAS unsigned*)(myws + WS_CNT + (size_t)b * 128);
    GAS unsigned* const tmo = (GAS unsigned*)(myws + WS_TMO);
    LAS unsigned* const misc = (LAS unsigned*)(lds + LDS_MISC);

    STAMP(0);
    char* const w16 = myws + WS_W16 + (size_t)ct * NKT * B_TILE;
    GAS unsigned* const wf = (GAS unsigned*)(myws + WS_WFLAGS) + ct * 256;
    if (wid >= 8) {
        if (puller) { BAR(); BAR(); return; }
        const int hw = wid - 8, pt = tid - NCONS;
        const int kt0 = label * 2 + q;
        const int colbase = p.rank * NS + ct * BN;
        const int colmax = p.rank * NS + NS - 4;
        const int kg = lane & 7;
        const int c1 = pt >> 3, c2 = 32 + hw;
        const bool xtra = lane < 8;
        const float* g1 = p.w + (size_t)(kt0 * BK + kg * 8) * NT + min(colbase + c1 * 4, colmax);
        const float* g2 = p.w + (size_t)(kt0 * BK + kg * 8) * NT + min(colbase + c2 * 4, colmax);
        f32x4 wa[8], xa[8];
        auto L = [&](f32x4 (&w)[8], f32x4 (&x)[8], int r) {
            const size_t o = (size_t)r * 16 * BK * NT;
#pragma unroll
            for (int e = 0; e < 8; ++e) w[e] = __builtin_nontemporal_load((const f32x4*)(g1 + o + (size_t)e * NT));
            if (xtra) {
#pragma unroll
                for (int e = 0; e < 8; ++e) x[e] = __builtin_nontemporal_load((const f32x4*)(g2 + o + (size_t)e * NT));
            }
        };
        auto S = [&](const f32x4 (&w)[8], const f32x4 (&x)[8], int r) {
            char* B = w16 + (size_t)(kt0 + 16 * r) * B_TILE;
#pragma unroll
            for (int i = 0; i < 4; ++i) {
                const int row = 4 * c1 + i;
                store_wt(B + row * 128 + ((kg ^ ((row >> 1) & 7)) << 4),
                         cvt8((f32x4){w[0][i], w[1][i], w[2][i], w[3][i]}, (f32x4){w[4][i], w[5][i], w[6][i], w[7][i]}));
            }
            if (xtra) {
#pragma unroll
                for (int i = 0; i < 4; ++i) {
                    const int row = 4 * c2 + i;
                    store_wt(B + row * 128 + ((kg ^ ((row >> 1) & 7)) << 4),
                             cvt8((f32x4){x[0][i], x[1][i], x[2][i], x[3][i]}, (f32x4){x[4][i], x[5][i], x[6][i], x[7][i]}));
                }
            }
        };
        unsigned call = 0;
        auto F = [&](int r) {
            WAIT_V(0);
            if (lane == 0) __hip_atomic_store(wf + (kt0 + 16 * r) * 4 + hw, call, __ATOMIC_RELAXED, __HIP_MEMORY_SCOPE_SYSTEM);
        };
        call = __builtin_amdgcn_readfirstlane(__hip_atomic_load(cnt, __ATOMIC_RELAXED, __HIP_MEMORY_SCOPE_AGENT)) + 1u;
        BAR();
        L(wa, xa, 0);
        S(wa, xa, 0);
        F(0);
        STAMP(7);
        BAR();
        if (misc[0] != 0u) return;
        int pos = -1;
        for (int r = 1; r < 4; ++r) {
            const int base = (r == 1 ? 0 : 16 * r - 17) + hw;
            for (; pos < base; ++pos) BAR();
            L(wa, xa, r);
            BAR(); BAR(); BAR();
            S(wa, xa, r);
            BAR(); BAR(); BAR();
            F(r);
            pos += 6;
            if (hw != 0 && r == 3) return;
            for (; pos < 16 * r - 5; ++pos) BAR();
            if (hw == 0) {
                typedef unsigned long long u64;
                GAS u64* fp = (GAS u64*)(wf + (16 * r + (lane & 15)) * 4 + (lane >> 4 & 1) * 2);
                u64 f;
                asm volatile("global_load_dwordx2 %0, %1, off sc0 sc1" : "=v"(f) : "v"(fp) : "memory");
                BAR(); BAR();
                asm volatile("s_waitcnt vmcnt(0)" : "+v"(f) :: "memory");
                if (!__all((unsigned)f >= call && (unsigned)(f >> 32) >= call)) {
                    const unsigned long long t0 = __builtin_amdgcn_s_memrealtime();
                    for (;;) {
                        f = __hip_atomic_load(fp, __ATOMIC_RELAXED, __HIP_MEMORY_SCOPE_SYSTEM);
                        if (__all((unsigned)f >= call && (unsigned)(f >> 32) >= call)) break;
                        if (__hip_atomic_load(tmo, __ATOMIC_RELAXED, __HIP_MEMORY_SCOPE_AGENT) == call ||
                            __builtin_amdgcn_s_memrealtime() - t0 > DEADLINE_TICKS) {
                            __hip_atomic_store(tmo, call, __ATOMIC_RELAXED, __HIP_MEMORY_SCOPE_AGENT);
                            break;
                        }
                        __builtin_amdgcn_s_sleep(1);
                    }
                    WAIT_V(0);
                }
            } else {
                BAR(); BAR();
            }
            BAR();
            pos += 3;
        }
        return;
    }
    unsigned call;
    size_t slot;
    GAS unsigned* fdst = nullptr;
    {
        const int pq = b >> 7, pt = (b >> 1) & 63, ph = b & 1;
        f32x4 v[2][2];
#pragma unroll
        for (int i = 0; i < 2; ++i) {
            const int idx = tid + i * NCONS, row = ph * 128 + (idx >> 3), ch = idx & 7;
            const float* g = p.x + (size_t)(pq * 256 + row) * KD + pt * BK + ch * 8;
            v[i][0] = *(const f32x4*)g;
            v[i][1] = *(const f32x4*)(g + 4);
        }
        if (wid == 0) fdst = (GAS unsigned*)(p.ws[lane & 7] + WS_FLAGS) + p.rank * GRID + b;
        const unsigned prev = __hip_atomic_load(cnt, __ATOMIC_RELAXED, __HIP_MEMORY_SCOPE_AGENT);
        asm volatile("" ::: "memory");
        call = __builtin_amdgcn_readfirstlane(prev) + 1u;
        slot = WS_X16 + (size_t)(call & 1u) * SLOT_BYTES;
        char* dst = myws + slot + (size_t)(pq * NKT + pt) * A_TILE;
#pragma unroll
        for (int i = 0; i < 2; ++i) {
            const int idx = tid + i * NCONS, row = ph * 128 + (idx >> 3), ch = idx & 7;
            store_wt(dst + row * 128 + ((ch ^ ((row >> 1) & 7)) << 4), cvt8(v[i][0], v[i][1]));
        }
    }
    WAIT_V(0);
    STAMP(1);
    BAR();
    if (wid == 0) {
        if (lane < NR) __hip_atomic_store(fdst, call, __ATOMIC_RELAXED, __HIP_MEMORY_SCOPE_SYSTEM);
        asm volatile("buffer_inv sc0 sc1" ::: "memory");
    }
    if (wid == 0) {
        typedef unsigned long long u64;
        GAS u64* fl = (GAS u64*)(myws + WS_FLAGS + ((size_t)src * GRID + q * 128) * 4) + lane;
        GAS u64* fw = (GAS u64*)(wf + (lane & 15) * 4 + (lane >> 4 & 1) * 2);
        const unsigned long long t0 = __builtin_amdgcn_s_memrealtime();
        for (;;) {
            const u64 f = __hip_atomic_load(fl, __ATOMIC_RELAXED, __HIP_MEMORY_SCOPE_SYSTEM);
            const u64 g = puller ? f : __hip_atomic_load(fw, __ATOMIC_RELAXED, __HIP_MEMORY_SCOPE_SYSTEM);
            const bool ok = (unsigned)f >= call && (unsigned)(f >> 32) >= call && (unsigned)g >= call && (unsigned)(g >> 32) >= call;
            if (__all(ok)) break;
            if (__hip_atomic_load(tmo, __ATOMIC_RELAXED, __HIP_MEMORY_SCOPE_AGENT) == call ||
                __builtin_amdgcn_s_memrealtime() - t0 > DEADLINE_TICKS) {
                __hip_atomic_store(tmo, call, __ATOMIC_RELAXED, __HIP_MEMORY_SCOPE_AGENT);
                break;
            }
            __builtin_amdgcn_s_sleep(1);
        }
        STAMP(2);
        WAIT_V(0);
        const unsigned bad = __hip_atomic_load(tmo, __ATOMIC_RELAXED, __HIP_MEMORY_SCOPE_AGENT) == call ? 1u : 0u;
        if (lane == 0) misc[0] = bad;
        WAIT_L(0);
        STAMP(3);
    }
    BAR();
    const bool bad = misc[0] != 0u;
    const char* asrc = p.ws[src] + slot + (size_t)q * NKT * A_TILE;
    unsigned voff[4];
#pragma unroll
    for (int i = 0; i < 4; ++i) voff[i] = (unsigned)tid * 16u + i * 8192u;

    if (puller) {
        if (!bad) {
            const unsigned l = lds0 + LDS_A + wid * 1024;
            LSTAMP_DECL;
            for (int t = 0; t < NKT; ++t) {
                const char* g = asrc + (size_t)t * A_TILE;
#pragma unroll
                for (int i = 0; i < 4; ++i) glds16(voff[i], g, l + i * 8192);
                WAIT_V(4);
                BAR();
                LSTAMP(t);
            }
            WAIT_V(0);
            LSTAMP_FLUSH();
        }
        STAMP(4);
    } else {
        const int fr = lane & 15, fq = lane >> 4;
        f32x4 acc[2][9];
#pragma unroll
        for (int i = 0; i < 2; ++i)
#pragma unroll
            for (int j = 0; j < 9; ++j) acc[i][j] = (f32x4){0.f, 0.f, 0.f, 0.f};

        if (!bad) {
            const int sw = (fr >> 1) & 7;
            const int aoff0 = LDS_A + (wid * 32 + fr) * 128 + ((fq ^ sw) << 4), aoff1 = aoff0 ^ 64;
            const int boff0 = LDS_B + fr * 128 + ((fq ^ sw) << 4), boff1 = boff0 ^ 64;
            const unsigned ldsa = lds0 + LDS_A + wid * 1024, ldsb = lds0 + LDS_B + wid * 1024;
            const bool third = wid < 2;
            f16x8 a0[2], a1[2], b0[3], b1[3];
#define LDA(dst, aoff, ab) do { _Pragma("unroll") for (int i_ = 0; i_ < 2; ++i_) dst[i_] = *(const LAS f16x8*)(lds + (ab) + (aoff) + i_ * 2048); } while (0)
#define LDB(dst, boff, bb, g) do { _Pragma("unroll") for (int j_ = 0; j_ < 3; ++j_) dst[j_] = *(const LAS f16x8*)(lds + (bb) + (boff) + ((g) * 3 + j_) * 2048); } while (0)
#define MMA(g, bf, af) do { _Pragma("unroll") for (int j_ = 0; j_ < 3; ++j_) _Pragma("unroll") for (int i_ = 0; i_ < 2; ++i_) \
        acc[i_][(g) * 3 + j_] = __builtin_amdgcn_mfma_f32_16x16x32_f16(bf[j_], af[i_], acc[i_][(g) * 3 + j_], 0, 0, 0); } while (0)
#pragma unroll
            for (int t = 0; t < 2; ++t) {
                if (third) glds16_sc(voff[2], w16 + (size_t)t * B_TILE, ldsb + t * B_TILE + 16384);
                glds16_sc(voff[0], w16 + (size_t)t * B_TILE, ldsb + t * B_TILE);
                glds16_sc(voff[1], w16 + (size_t)t * B_TILE, ldsb + t * B_TILE + 8192);
#pragma unroll
                for (int i = 0; i < 4; ++i) glds16(voff[i], asrc + (size_t)t * A_TILE, ldsa + t * A_TILE + i * 8192);
            }
            if (third) glds16_sc(voff[2], w16 + (size_t)2 * B_TILE, ldsb + 2 * B_TILE + 16384);
            glds16_sc(voff[0], w16 + (size_t)2 * B_TILE, ldsb + 2 * B_TILE);
            if (third) WAIT_V(9); else WAIT_V(7);
            BAR();
            LDA(a0, aoff0, 0);
            LDB(b0, boff0, 0, 0);
            LSTAMP_DECL;
            int buf = 0;
            auto iter = [&](auto mode_, int t) {
                constexpr int MODE = decltype(mode_)::value;
                const int bn = (buf == 2) ? 0 : buf + 1;
                const int bp = (buf == 0) ? 2 : buf - 1;
                const int ab = buf * A_TILE, bb = buf * B_TILE;
                const char* ga = asrc + (size_t)(t + 2) * A_TILE;
                const char* gb = w16 + (size_t)(t + 2) * B_TILE;
                const unsigned la = ldsa + bp * A_TILE, lb = ldsb + bp * B_TILE;
                SCHED();
                LDB(b1, boff0, bb, 1); LDA(a1, aoff1, ab);
                SCHED();
                MMA(0, b0, a0);
                if (MODE <= 1) glds16_sc(voff[1], gb, lb + 8192);
                SCHED();
                LDB(b0, boff0, bb, 2);
                SCHED();
                MMA(1, b1, a0);
                if (MODE <= 1) glds16(voff[0], ga, la);
                SCHED();
                LDB(b1, boff1, bb, 0);
                SCHED();
                MMA(2, b0, a0);
                if (MODE <= 1) glds16(voff[1], ga, la + 8192);
                SCHED();
                LDB(b0, boff1, bb, 1);
                SCHED();
                MMA(0, b1, a1);
                if (MODE <= 1) glds16(voff[2], ga, la + 2 * 8192);
                SCHED();
                LDB(b1, boff1, bb, 2);
                SCHED();
                MMA(1, b0, a1);
                if (MODE <= 1) glds16(voff[3], ga, la + 3 * 8192);
                SCHED();
                if (MODE <= 2) {
                    if (MODE <= 1) { if (third) WAIT_V(7); else WAIT_V(6); } else WAIT_V(0);
                    WAIT_L(0);
                    LSTAMP(t);
                    BAR();
                    LDA(a0, aoff0, bn * A_TILE);
                    LDB(b0, boff0, bn * B_TILE, 0);
                    SCHED();
                }
                MMA(2, b1, a1);
                if (MODE == 0) {
                    if (third) glds16_sc(voff[2], gb + B_TILE, ldsb + bb + 16384);
                    glds16_sc(voff[0], gb + B_TILE, ldsb + bb);
                }
                buf = bn;
            };
            for (int t = 0; t < NKT - 3; ++t) iter(IC<0>{}, t);
            iter(IC<1>{}, NKT - 3);
            iter(IC<2>{}, NKT - 2);
            iter(IC<3>{}, NKT - 1);
            LSTAMP_FLUSH();
        }
        STAMP(4);
        const float qnan = __builtin_nanf("");
        const bool badf = bad || __hip_atomic_load(tmo, __ATOMIC_RELAXED, __HIP_MEMORY_SCOPE_AGENT) == call;
        float* o = p.out + (size_t)(src * MS + q * BM + wid * 32 + fr) * NS + ct * BN + fq * 4;
        const int nvalid = (NS - ct * BN) >> 4;
#pragma unroll
        for (int i = 0; i < 2; ++i)
#pragma unroll
            for (int j = 0; j < 9; ++j) {
                f32x4 v = acc[i][j];
                v[0] = fmaxf(v[0], 0.f); v[1] = fmaxf(v[1], 0.f); v[2] = fmaxf(v[2], 0.f); v[3] = fmaxf(v[3], 0.f);
                if (badf) v = (f32x4){qnan, qnan, qnan, qnan};
                if (j < nvalid) *(f32x4*)(o + (size_t)i * 16 * NS + j * 16) = v;
            }
        STAMP(5);
    }
    if (tid == 0) __hip_atomic_store(cnt, call, __ATOMIC_RELAXED, __HIP_MEMORY_SCOPE_AGENT);
}

}

extern "C" void kernel_launch(int rank, int nranks, ncclComm_t comm,
                              hipStream_t stream,
                              void* const* d_in, const int* in_sizes, int n_in,
                              void* d_out, int out_size,
                              const int* in_dtypes, int out_dtype,
                              void* const* d_out_all,
                              void* const* d_ws_all, size_t ws_bytes) {
    (void)comm; (void)in_dtypes; (void)out_dtype; (void)d_out_all;
    if (nranks != NR || n_in < 2 || (unsigned)rank >= (unsigned)NR || ws_bytes < WS_NEED ||
        in_sizes[0] != MS * KD || in_sizes[1] != KD * NT || out_size != NR * MS * NS) {
        fprintf(stderr, "kernel_launch: unexpected shapes\n");
        return;
    }
    static bool attr_set[64] = {};
    int dev = 0;
    HIP_CHECK(hipGetDevice(&dev));
    if (dev >= 0 && dev < 64 && !attr_set[dev]) {
        HIP_CHECK(hipFuncSetAttribute((const void*)gemm_pull_kernel, hipFuncAttributeMaxDynamicSharedMemorySize, LDS_BYTES));
        attr_set[dev] = true;
    }
    Params p;
    p.x = (const float*)d_in[0];
    p.w = (const float*)d_in[1];
    p.out = (float*)d_out;
    for (int i = 0; i < NR; ++i) p.ws[i] = (char*)d_ws_all[i];
    p.rank = rank;
    p.pad = 0;
    gemm_pull_kernel<<<GRID, NTHREADS, LDS_BYTES, stream>>>(p);
    HIP_CHECK(hipGetLastError());
}
```

```cpp
#include <hip/hip_runtime.h>
#include <rccl/rccl.h>
#include <cstdio>

#define HIP_CHECK(expr) do { hipError_t _e = (expr); if (_e != hipSuccess) { \
    fprintf(stderr, "HIP error %s at %s:%d\n", hipGetErrorString(_e), __FILE__, __LINE__); \
    return; } } while (0)

namespace {

typedef _Float16 f16x2 __attribute__((ext_vector_type(2)));
typedef _Float16 f16x8 __attribute__((ext_vector_type(8)));
typedef float f32x2 __attribute__((ext_vector_type(2)));
typedef float f32x4 __attribute__((ext_vector_type(4)));
typedef unsigned u32x4 __attribute__((ext_vector_type(4)));
typedef unsigned long long u64;
#define LAS __attribute__((address_space(3)))
#define GAS __attribute__((address_space(1)))

constexpr int NR = 8;
constexpr int MS = 512;
constexpr int KD = 4096;
constexpr int NT = 16384;
constexpr int NS = 2048;
constexpr int BM = 256, BN = 144, BK = 64;
constexpr int NCT = 15;
constexpr int NKT = KD / BK;
constexpr int A_TILE = BM * BK * 2;
constexpr int B_TILE = BN * BK * 2;
constexpr int NBUF = 3;
constexpr int LDS_A = 0;
constexpr int LDS_B = NBUF * A_TILE;
constexpr int LDS_MISC = LDS_B + NBUF * B_TILE;
constexpr int LDS_BYTES = LDS_MISC + 64;
constexpr int GRID = 256;
constexpr int NCONS = 512;
constexpr int NTHREADS = 768;
#define PULL_LAG 8

constexpr size_t SLOT_BYTES = (size_t)MS * KD * 2;
constexpr size_t WS_X16 = 0;
constexpr size_t WS_FLAGS = 2 * SLOT_BYTES;
constexpr size_t WS_XCNT = WS_FLAGS + (size_t)NR * 2 * 128;
constexpr size_t WS_INV = WS_XCNT + 16 * 128;
constexpr size_t WS_WFLAGS = WS_INV + 32 * 128;
constexpr size_t WS_CNT = WS_WFLAGS + 16 * 64 * 4 * 4;
constexpr size_t WS_PROG = WS_CNT + (size_t)GRID * 128;
constexpr size_t WS_TMO = WS_PROG + (size_t)GRID * 128;
constexpr size_t WS_W16 = 9u << 20;
constexpr size_t WS_NEED = WS_W16 + (size_t)NCT * NKT * B_TILE;
static_assert(WS_TMO + 128 <= WS_W16, "workspace layout");

constexpr u64 DEADLINE_TICKS = 30000000ull;

struct Params {
    const float* x;
    const float* w;
    float* out;
    char* ws[NR];
    int rank;
    int pad;
};

template <int V> struct IC { static constexpr int value = V; };

__device__ __forceinline__ f16x8 cvt8(f32x4 a, f32x4 b) {
    f16x2 p0 = __builtin_convertvector((f32x2){a[0], a[1]}, f16x2);
    f16x2 p1 = __builtin_convertvector((f32x2){a[2], a[3]}, f16x2);
    f16x2 p2 = __builtin_convertvector((f32x2){b[0], b[1]}, f16x2);
    f16x2 p3 = __builtin_convertvector((f32x2){b[2], b[3]}, f16x2);
    return (f16x8){p0[0], p0[1], p1[0], p1[1], p2[0], p2[1], p3[0], p3[1]};
}

__device__ __forceinline__ void glds16(unsigned voff, const char* sbase, unsigned ldsaddr) {
    asm volatile("s_mov_b32 m0, %2\n\ts_nop 0\n\tglobal_load_lds_dwordx4 %0, %1" :: "v"(voff), "s"(sbase), "s"(ldsaddr) : "memory");
}
__device__ __forceinline__ void glds16_sc(unsigned voff, const char* sbase, unsigned ldsaddr) {
    asm volatile("s_mov_b32 m0, %2\n\ts_nop 0\n\tglobal_load_lds_dwordx4 %0, %1 sc0 sc1" :: "v"(voff), "s"(sbase), "s"(ldsaddr) : "memory");
}
__device__ __forceinline__ void store_wt(void* d, f16x8 h) {
    asm volatile("global_store_dwordx4 %0, %1, off sc0 sc1\n\ts_nop 1" :: "v"(d), "v"(__builtin_bit_cast(u32x4, h)) : "memory");
}
__device__ __forceinline__ void store_u32_wt(GAS unsigned* d, unsigned v) {
    asm volatile("global_store_dword %0, %1, off sc0 sc1\n\ts_nop 1" :: "v"(d), "v"(v) : "memory");
}
template <int N> __device__ __forceinline__ void wait_v() {
    static_assert(N >= 0 && N < 64, "vmcnt");
    asm volatile("s_waitcnt vmcnt(%0)" :: "i"(N) : "memory");
}

#define WAIT_V(n) asm volatile("s_waitcnt vmcnt(" #n ")" ::: "memory")
#define WAIT_L(n) asm volatile("s_waitcnt lgkmcnt(" #n ")" ::: "memory")
#define BAR() do { asm volatile("" ::: "memory"); __builtin_amdgcn_s_barrier(); asm volatile("" ::: "memory"); } while (0)
#define SCHED() __builtin_amdgcn_sched_barrier(0)

#define STAMP(i) do {} while (0)
#define LSTAMP_DECL do {} while (0)
#define LSTAMP(t) do {} while (0)
#define CSTAMP(i) do {} while (0)
#define LSTAMP_FLUSH() do {} while (0)

template <class P> __device__ __forceinline__ void spin_flags(GAS u64* fp, GAS u64* fp2, GAS unsigned* tmo, unsigned call, P ok) {
    const u64 t0 = __builtin_amdgcn_s_memrealtime();
    for (;;) {
        const u64 f = __hip_atomic_load(fp, __ATOMIC_RELAXED, __HIP_MEMORY_SCOPE_SYSTEM);
        const u64 g = __hip_atomic_load(fp2, __ATOMIC_RELAXED, __HIP_MEMORY_SCOPE_SYSTEM);
        if (__all(ok(f) && ok(g))) break;
        if (__hip_atomic_load(tmo, __ATOMIC_RELAXED, __HIP_MEMORY_SCOPE_AGENT) == call ||
            __builtin_amdgcn_s_memrealtime() - t0 > DEADLINE_TICKS) {
            __hip_atomic_store(tmo, call, __ATOMIC_RELAXED, __HIP_MEMORY_SCOPE_AGENT);
            break;
        }
        __builtin_amdgcn_s_sleep(1);
    }
    WAIT_V(0);
}

__device__ __forceinline__ bool spin_words(GAS unsigned* fp, GAS unsigned* tmo, unsigned call) {
    const u64 t0 = __builtin_amdgcn_s_memrealtime();
    for (;;) {
        const unsigned f = __hip_atomic_load(fp, __ATOMIC_RELAXED, __HIP_MEMORY_SCOPE_SYSTEM);
        if (__all(f >= call)) return false;
        if (__hip_atomic_load(tmo, __ATOMIC_RELAXED, __HIP_MEMORY_SCOPE_AGENT) == call ||
            __builtin_amdgcn_s_memrealtime() - t0 > DEADLINE_TICKS) {
            __hip_atomic_store(tmo, call, __ATOMIC_RELAXED, __HIP_MEMORY_SCOPE_AGENT);
            return true;
        }
        __builtin_amdgcn_s_sleep(1);
    }
}

__global__ void __launch_bounds__(NTHREADS, 3) gemm_pull_kernel(Params p) {
    extern __shared__ __attribute__((aligned(16))) char smem[];
    LAS char* lds = (LAS char*)smem;
    const unsigned lds0 = (unsigned)(size_t)lds;
    const int tid = threadIdx.x;
    const int wid = __builtin_amdgcn_readfirstlane(tid >> 6);
    const int lane = tid & 63;
    const int b = blockIdx.x;
    const int label = b & 7, rr = b >> 3;
    const bool puller = rr >= 2 * NCT;
    const int q = puller ? rr - 2 * NCT : rr / NCT;
    const int ct = puller ? 0 : rr - q * NCT;
    const unsigned src = (unsigned)(p.rank + label) & 7u;
    char* const myws = p.ws[p.rank];
    GAS unsigned* const cnt = (GAS unsigned*)(myws + WS_CNT + (size_t)b * 128);
    GAS unsigned* const tmo = (GAS unsigned*)(myws + WS_TMO);
    LAS unsigned* const misc = (LAS unsigned*)(lds + LDS_MISC);
    char* const w16 = myws + WS_W16 + (size_t)ct * NKT * B_TILE;
    GAS unsigned* const wf = (GAS unsigned*)(myws + WS_WFLAGS) + ct * 256;

    GAS unsigned* const invw = (GAS unsigned*)(myws + WS_INV + (size_t)((unsigned)__builtin_amdgcn_s_getreg((3 << 11) | 20) & 0xFu) * 256);

    STAMP(0);
    if (wid >= 8) {
        const int lw = wid - 8, pt = tid - NCONS;
        const unsigned call = __builtin_amdgcn_readfirstlane(__hip_atomic_load(cnt, __ATOMIC_RELAXED, __HIP_MEMORY_SCOPE_AGENT)) + 1u;
        unsigned won = 0;
        if (lw == 0) {
            if (lane == 0) won = __hip_atomic_fetch_max(invw, call, __ATOMIC_RELAXED, __HIP_MEMORY_SCOPE_AGENT) < call ? 1u : 0u;
            won = __builtin_amdgcn_readfirstlane(won);
        }
        BAR();
        if (won) asm volatile("buffer_inv sc0 sc1" ::: "memory");
        BAR();
        if (won) {
            WAIT_V(0);
            if (lane == 0) __hip_atomic_store(invw + 32, call, __ATOMIC_RELAXED, __HIP_MEMORY_SCOPE_AGENT);
        }
        BAR();
        if (puller) { BAR(); return; }
        const int kt0 = label * 2 + q;
        const int colbase = p.rank * NS + ct * BN;
        const int colmax = p.rank * NS + NS - 4;
        const int kg = lane & 7;
        const int c1 = pt >> 3, c2 = 32 + lw;
        const float* g1 = p.w + (size_t)(kt0 * BK + kg * 8) * NT + min(colbase + c1 * 4, colmax);
        const float* g2 = p.w + (size_t)(kt0 * BK + kg * 8) * NT + min(colbase + c2 * 4, colmax);
        f32x4 wa[8], xa[8];
        auto WL = [&](int r) {
            const size_t o = (size_t)r * 16 * BK * NT;
#pragma unroll
            for (int e = 0; e < 8; ++e) asm volatile("global_load_dwordx4 %0, %1, off nt" : "=v"(wa[e]) : "v"(g1 + o + (size_t)e * NT) : "memory");
            const float* h = g2 + o;
            asm volatile("s_mov_b64 exec, 0xff\n\t"
                         "global_load_dwordx4 %0, %8, off nt\n\tglobal_load_dwordx4 %1, %9, off nt\n\t"
                         "global_load_dwordx4 %2, %10, off nt\n\tglobal_load_dwordx4 %3, %11, off nt\n\t"
                         "global_load_dwordx4 %4, %12, off nt\n\tglobal_load_dwordx4 %5, %13, off nt\n\t"
                         "global_load_dwordx4 %6, %14, off nt\n\tglobal_load_dwordx4 %7, %15, off nt\n\t"
                         "s_mov_b64 exec, -1"
                         : "=&v"(xa[0]), "=&v"(xa[1]), "=&v"(xa[2]), "=&v"(xa[3]), "=&v"(xa[4]), "=&v"(xa[5]), "=&v"(xa[6]), "=&v"(xa[7])
                         : "v"(h), "v"(h + NT), "v"(h + 2 * NT), "v"(h + 3 * NT), "v"(h + 4 * NT), "v"(h + 5 * NT), "v"(h + 6 * NT), "v"(h + 7 * NT)
                         : "memory");
        };
        auto W_USE = [&]() {
            asm volatile("" : "+v"(wa[0]), "+v"(wa[1]), "+v"(wa[2]), "+v"(wa[3]), "+v"(wa[4]), "+v"(wa[5]), "+v"(wa[6]), "+v"(wa[7]),
                              "+v"(xa[0]), "+v"(xa[1]), "+v"(xa[2]), "+v"(xa[3]), "+v"(xa[4]), "+v"(xa[5]), "+v"(xa[6]), "+v"(xa[7]) :: "memory");
        };
        auto WS = [&](int r) {
            char* B = w16 + (size_t)(kt0 + 16 * r) * B_TILE;
            u32x4 h2[4];
            char* d2[4];
#pragma unroll
            for (int i = 0; i < 4; ++i) {
                const int row = 4 * c1 + i, row2 = 4 * c2 + i;
                store_wt(B + row * 128 + ((kg ^ ((row >> 1) & 7)) << 4),
                         cvt8((f32x4){wa[0][i], wa[1][i], wa[2][i], wa[3][i]}, (f32x4){wa[4][i], wa[5][i], wa[6][i], wa[7][i]}));
                h2[i] = __builtin_bit_cast(u32x4, cvt8((f32x4){xa[0][i], xa[1][i], xa[2][i], xa[3][i]}, (f32x4){xa[4][i], xa[5][i], xa[6][i], xa[7][i]}));
                d2[i] = B + row2 * 128 + ((kg ^ ((row2 >> 1) & 7)) << 4);
            }
            asm volatile("s_mov_b64 exec, 0xff\n\t"
                         "global_store_dwordx4 %0, %4, off sc0 sc1\n\tglobal_store_dwordx4 %1, %5, off sc0 sc1\n\t"
                         "global_store_dwordx4 %2, %6, off sc0 sc1\n\tglobal_store_dwordx4 %3, %7, off sc0 sc1\n\t"
                         "s_nop 1\n\ts_mov_b64 exec, -1"
                         :: "v"(d2[0]), "v"(d2[1]), "v"(d2[2]), "v"(d2[3]), "v"(h2[0]), "v"(h2[1]), "v"(h2[2]), "v"(h2[3]) : "memory");
        };
        const size_t slot = WS_X16 + (size_t)(call & 1u) * SLOT_BYTES;
        WL(0);
        WAIT_V(0);
        W_USE();
        WS(0);
        WAIT_V(0);
        if (lane == 0) store_u32_wt(wf + kt0 * 4 + lw, call);
        STAMP(7);
        BAR();
        if (misc[0] != 0u) return;

        const char* asrc = p.ws[src] + slot + (size_t)q * NKT * A_TILE;
        const unsigned voff = (unsigned)lane * 16u + (unsigned)lw * 1024u;
        unsigned vo[8];
#pragma unroll
        for (int i = 0; i < 8; ++i) vo[i] = voff + i * 4096u;
        const unsigned la0 = lds0 + LDS_A + lw * 1024, lb0 = lds0 + LDS_B + lw * 1024;
        GAS unsigned* const prog = (GAS unsigned*)(myws + WS_PROG + (size_t)b * 128);
        const unsigned pbase = call * 64u;
        auto tile = [&](auto nb_, int T, int buf) {
            constexpr int NB = decltype(nb_)::value;
            const char* gb = w16 + (size_t)T * B_TILE;
            const char* ga = asrc + (size_t)T * A_TILE;
#pragma unroll
            for (int i = 0; i < NB; ++i) glds16_sc(vo[i], gb, lb0 + buf * B_TILE + i * 4096);
#pragma unroll
            for (int i = 0; i < 8; ++i) glds16(vo[i], ga, la0 + buf * A_TILE + i * 4096);
        };
        auto loop = [&](auto nb_) {
            constexpr int NB = decltype(nb_)::value;
            constexpr int N = NB + 8;
            constexpr int XP = (NB == 4) ? 1 : 0;
            int t = 0, buf = 2;
            auto step = [&](auto x_, auto&& extra) {
                constexpr int X = decltype(x_)::value;
                tile(nb_, t + 2, buf);
                if (XP) store_u32_wt(prog, pbase + (unsigned)t - 1u);
                extra();
                wait_v<N + XP + X>();
                BAR();
                ++t; buf = (buf == 2) ? 0 : buf + 1;
            };
            auto none = [&]() {};
            tile(nb_, 0, 0); tile(nb_, 1, 1); tile(nb_, 2, 2);
            wait_v<2 * N>();
            BAR();
            wait_v<N>();
            BAR();
            t = 1; buf = 0;
            GAS u64* const fp = (GAS u64*)(wf + (lane & 15) * 4 + (lane >> 4 & 1) * 2);
#pragma unroll 1
            for (int r = 1; r < 4; ++r) {
                const int base = (r == 1 ? 1 : 16 * r - 17) + lw;
                while (t < base) step(IC<0>{}, none);
                step(IC<16>{}, [&]() { WL(r); });
                step(IC<0>{}, none);
                step(IC<0>{}, none);
                W_USE();
                step(IC<8>{}, [&]() { WS(r); });
                step(IC<0>{}, none);
                step(IC<0>{}, none);
                step(IC<1>{}, [&]() { if (lane == 0) store_u32_wt(wf + (kt0 + 16 * r) * 4 + lw, call); });
                while (t < 16 * r - 5) step(IC<0>{}, none);
                if (lw == 0) {
                    u64 f;
                    GAS u64* const fr_ = fp + r * 32;
                    step(IC<1>{}, [&]() { asm volatile("global_load_dwordx2 %0, %1, off sc0 sc1" : "=v"(f) : "v"(fr_) : "memory"); });
                    step(IC<0>{}, none);
                    asm volatile("" : "+v"(f) :: "memory");
                    if (!__all((unsigned)f >= call && (unsigned)(f >> 32) >= call))
                        spin_flags(fr_, fr_, tmo, call, [&](u64 v) { return (unsigned)v >= call && (unsigned)(v >> 32) >= call; });
                    step(IC<0>{}, none);
                } else {
                    step(IC<0>{}, none); step(IC<0>{}, none); step(IC<0>{}, none);
                }
            }
            while (t < NKT - 2) step(IC<0>{}, none);
            WAIT_V(0);
            BAR();
        };
        if (lw < 2) loop(IC<5>{}); else loop(IC<4>{});
        return;
    }

    unsigned call;
    size_t slot;
    GAS unsigned* fdst = nullptr;
    {
        const int pq = b >> 7, pt = (b >> 1) & 63, ph = b & 1;
        f32x4 v[2][2];
#pragma unroll
        for (int i = 0; i < 2; ++i) {
            const int idx = tid + i * NCONS, row = ph * 128 + (idx >> 3), ch = idx & 7;
            const float* g = p.x + (size_t)(pq * 256 + row) * KD + pt * BK + ch * 8;
            v[i][0] = *(const f32x4*)g;
            v[i][1] = *(const f32x4*)(g + 4);
        }
        if (wid == 0) fdst = (GAS unsigned*)(p.ws[lane & 7] + WS_FLAGS + (size_t)(p.rank * 2 + (b >> 7)) * 128) + label;
        const unsigned prev = __hip_atomic_load(cnt, __ATOMIC_RELAXED, __HIP_MEMORY_SCOPE_AGENT);
        asm volatile("" ::: "memory");
        call = __builtin_amdgcn_readfirstlane(prev) + 1u;
        slot = WS_X16 + (size_t)(call & 1u) * SLOT_BYTES;
        char* dst = myws + slot + (size_t)(pq * NKT + pt) * A_TILE;
#pragma unroll
        for (int i = 0; i < 2; ++i) {
            const int idx = tid + i * NCONS, row = ph * 128 + (idx >> 3), ch = idx & 7;
            store_wt(dst + row * 128 + ((ch ^ ((row >> 1) & 7)) << 4), cvt8(v[i][0], v[i][1]));
        }
    }
    BAR();
    WAIT_V(0);
    STAMP(1);
    BAR();
    if (wid == 0) {
        GAS unsigned* xc = (GAS unsigned*)(myws + WS_XCNT + (size_t)((b >> 7) * 8 + label) * 128);
        unsigned old = 0;
        if (lane == 0) old = __hip_atomic_fetch_add(xc, 1u, __ATOMIC_RELAXED, __HIP_MEMORY_SCOPE_AGENT);
        old = __builtin_amdgcn_readfirstlane(old);
        if (old + 1u == call * 16u && lane < NR) __hip_atomic_store(fdst, call, __ATOMIC_RELAXED, __HIP_MEMORY_SCOPE_SYSTEM);
    }
    BAR();
    if (wid == 0) {
        STAMP(100);
        GAS unsigned* fp = lane < 8 ? (GAS unsigned*)(myws + WS_FLAGS + (size_t)(src * 2 + q) * 128) + lane : invw + 32;
        unsigned bad = spin_words(fp, tmo, call) ? 1u : 0u;
        STAMP(101);
        if (!puller) {
            GAS u64* fw = (GAS u64*)(wf + (lane & 15) * 4 + (lane >> 4 & 1) * 2);
            spin_flags(fw, fw, tmo, call, [&](u64 v) { return (unsigned)v >= call && (unsigned)(v >> 32) >= call; });
            bad = __hip_atomic_load(tmo, __ATOMIC_RELAXED, __HIP_MEMORY_SCOPE_AGENT) == call ? 1u : 0u;
        }
        WAIT_V(0);
        STAMP(2);
        if (lane == 0) misc[0] = bad;
        WAIT_L(0);
        STAMP(3);
    }
    BAR();
    const bool bad = misc[0] != 0u;

    if (puller) {
        if (!bad) {
            const char* asrc = p.ws[src] + slot + (size_t)q * NKT * A_TILE;
            const unsigned l = lds0 + LDS_A + wid * 1024;
            const unsigned* prog = (const unsigned*)(myws + WS_PROG + (size_t)(q * NCT * 8 + label) * 128);
            const unsigned pbase = call * 64u;
            unsigned voff[4];
#pragma unroll
            for (int i = 0; i < 4; ++i) voff[i] = (unsigned)tid * 16u + i * 8192u;
            unsigned seen = 0;
            LSTAMP_DECL;
            for (int t = 0; t < NKT; ++t) {
                if (t >= PULL_LAG && wid == 0) {
                    const unsigned need = pbase + (unsigned)(t - PULL_LAG);
                    WAIT_L(0);
                    asm volatile("" : "+s"(seen));
                    if ((int)(seen - need) < 0) {
                        const u64 t0 = __builtin_amdgcn_s_memrealtime();
                        do {
                            seen = __builtin_amdgcn_readfirstlane(__hip_atomic_load((GAS unsigned*)prog, __ATOMIC_RELAXED, __HIP_MEMORY_SCOPE_SYSTEM));
                            if (__builtin_amdgcn_s_memrealtime() - t0 > DEADLINE_TICKS) break;
                            __builtin_amdgcn_s_sleep(2);
                        } while ((int)(seen - need) < 0);
                    }
                }
                if (t >= PULL_LAG) BAR();
                if (t + 1 >= PULL_LAG && wid == 0) asm volatile("s_load_dword %0, %1, 0x0 glc" : "=s"(seen) : "s"(prog) : "memory");
                const char* g = asrc + (size_t)t * A_TILE;
#pragma unroll
                for (int i = 0; i < 4; ++i) glds16(voff[i], g, l + i * 8192);
                WAIT_V(4);
                BAR();
                LSTAMP(t);
            }
            WAIT_V(0);
            WAIT_L(0);
            LSTAMP_FLUSH();
        }
        STAMP(4);
    } else {
        const int fr = lane & 15, fq = lane >> 4;
        f32x4 acc[2][9];
#pragma unroll
        for (int i = 0; i < 2; ++i)
#pragma unroll
            for (int j = 0; j < 9; ++j) acc[i][j] = (f32x4){0.f, 0.f, 0.f, 0.f};

        if (!bad) {
            const int sw = (fr >> 1) & 7;
            const int aoff0 = LDS_A + (wid * 32 + fr) * 128 + ((fq ^ sw) << 4), aoff1 = aoff0 ^ 64;
            const int boff0 = LDS_B + fr * 128 + ((fq ^ sw) << 4), boff1 = boff0 ^ 64;
            f16x8 a0[2], a1[2], b0[3], b1[3];
#define LDA(dst, aoff, ab) do { _Pragma("unroll") for (int i_ = 0; i_ < 2; ++i_) dst[i_] = *(const LAS f16x8*)(lds + (ab) + (aoff) + i_ * 2048); } while (0)
#define LDB(dst, boff, bb, g) do { _Pragma("unroll") for (int j_ = 0; j_ < 3; ++j_) dst[j_] = *(const LAS f16x8*)(lds + (bb) + (boff) + ((g) * 3 + j_) * 2048); } while (0)
#define MMA(g, bf, af) do { _Pragma("unroll") for (int j_ = 0; j_ < 3; ++j_) _Pragma("unroll") for (int i_ = 0; i_ < 2; ++i_) \
        acc[i_][(g) * 3 + j_] = __builtin_amdgcn_mfma_f32_16x16x32_f16(bf[j_], af[i_], acc[i_][(g) * 3 + j_], 0, 0, 0); } while (0)
            BAR();
            LDA(a0, aoff0, 0);
            LDB(b0, boff0, 0, 0);
            LSTAMP_DECL;
            CSTAMP(0);
            int buf = 0;
            auto iter = [&](auto last_, int t) {
                constexpr bool LAST = decltype(last_)::value != 0;
                const int bn = (buf == 2) ? 0 : buf + 1;
                const int ab = buf * A_TILE, bb = buf * B_TILE;
                SCHED();
                LDB(b1, boff0, bb, 1); LDA(a1, aoff1, ab);
                SCHED();
                MMA(0, b0, a0);
                SCHED();
                LDB(b0, boff0, bb, 2);
                SCHED();
                MMA(1, b1, a0);
                SCHED();
                LDB(b1, boff1, bb, 0);
                SCHED();
                MMA(2, b0, a0);
                SCHED();
                LDB(b0, boff1, bb, 1);
                SCHED();
                MMA(0, b1, a1);
                SCHED();
                LDB(b1, boff1, bb, 2);
                SCHED();
                MMA(1, b0, a1);
                SCHED();
                if (!LAST) {
                    WAIT_L(0);
                    LSTAMP(t);
                    BAR();
                    LDA(a0, aoff0, bn * A_TILE);
                    LDB(b0, boff0, bn * B_TILE, 0);
                    SCHED();
                }
                MMA(2, b1, a1);
                buf = bn;
            };
            for (int t = 0; t < NKT - 1; ++t) iter(IC<0>{}, t);
            iter(IC<1>{}, NKT - 1);
            CSTAMP(1);
            LSTAMP_FLUSH();
        }
        STAMP(4);
        const float qnan = __builtin_nanf("");
        const bool badf = bad || __hip_atomic_load(tmo, __ATOMIC_RELAXED, __HIP_MEMORY_SCOPE_AGENT) == call;
        float* o = p.out + (size_t)(src * MS + q * BM + wid * 32 + fr) * NS + ct * BN + fq * 4;
        const int nvalid = (NS - ct * BN) >> 4;
#pragma unroll
        for (int i = 0; i < 2; ++i)
#pragma unroll
            for (int j = 0; j < 9; ++j) {
                f32x4 v = acc[i][j];
                v[0] = v[0] < 0.f ? 0.f : v[0]; v[1] = v[1] < 0.f ? 0.f : v[1]; v[2] = v[2] < 0.f ? 0.f : v[2]; v[3] = v[3] < 0.f ? 0.f : v[3];
                if (badf) v = (f32x4){qnan, qnan, qnan, qnan};
                if (j < nvalid) *(f32x4*)(o + (size_t)i * 16 * NS + j * 16) = v;
            }
        STAMP(5);
    }
    if (tid == 0) __hip_atomic_store(cnt, call, __ATOMIC_RELAXED, __HIP_MEMORY_SCOPE_AGENT);
}

}

extern "C" void kernel_launch(int rank, int nranks, ncclComm_t comm,
                              hipStream_t stream,
                              void* const* d_in, const int* in_sizes, int n_in,
                              void* d_out, int out_size,
                              const int* in_dtypes, int out_dtype,
                              void* const* d_out_all,
                              void* const* d_ws_all, size_t ws_bytes) {
    (void)comm; (void)in_dtypes; (void)out_dtype; (void)d_out_all;
    if (nranks != NR || n_in < 2 || (unsigned)rank >= (unsigned)NR || ws_bytes < WS_NEED ||
        in_sizes[0] != MS * KD || in_sizes[1] != KD * NT || out_size != NR * MS * NS) {
        fprintf(stderr, "kernel_launch: unexpected shapes\n");
        return;
    }
    static bool attr_set[64] = {};
    int dev = 0;
    HIP_CHECK(hipGetDevice(&dev));
    if (dev >= 0 && dev < 64 && !attr_set[dev]) {
        HIP_CHECK(hipFuncSetAttribute((const void*)gemm_pull_kernel, hipFuncAttributeMaxDynamicSharedMemorySize, LDS_BYTES));
        attr_set[dev] = true;
    }
    Params p;
    p.x = (const float*)d_in[0];
    p.w = (const float*)d_in[1];
    p.out = (float*)d_out;
    for (int i = 0; i < NR; ++i) p.ws[i] = (char*)d_ws_all[i];
    p.rank = rank;
    p.pad = 0;
    gemm_pull_kernel<<<GRID, NTHREADS, LDS_BYTES, stream>>>(p);
    HIP_CHECK(hipGetLastError());
}
```

```cpp
#include <hip/hip_runtime.h>
#include <rccl/rccl.h>
#include <cstdio>

#define HIP_CHECK(expr) do { hipError_t _e = (expr); if (_e != hipSuccess) { \
    fprintf(stderr, "HIP error %s at %s:%d\n", hipGetErrorString(_e), __FILE__, __LINE__); \
    return; } } while (0)

namespace {

typedef _Float16 f16x2 __attribute__((ext_vector_type(2)));
typedef _Float16 f16x8 __attribute__((ext_vector_type(8)));
typedef float f32x2 __attribute__((ext_vector_type(2)));
typedef float f32x4 __attribute__((ext_vector_type(4)));
typedef unsigned u32x2 __attribute__((ext_vector_type(2)));
typedef unsigned u32x4 __attribute__((ext_vector_type(4)));
typedef unsigned long long u64;
#define LAS __attribute__((address_space(3)))
#define GAS __attribute__((address_space(1)))

constexpr int NR = 8;
constexpr int MS = 512;
constexpr int KD = 4096;
constexpr int NT = 16384;
constexpr int NS = 2048;
constexpr int BM = 256, BN = 144, BK = 64;
constexpr int NCT = 15;
constexpr int NKT = KD / BK;
constexpr int A_TILE = BM * BK * 2;
constexpr int B_TILE = BN * BK * 2;
constexpr int NBUF = 3;
constexpr int LDS_A = 0;
constexpr int LDS_B = NBUF * A_TILE;
constexpr int LDS_MISC = LDS_B + NBUF * B_TILE;
constexpr int LDS_BYTES = LDS_MISC + 128;
constexpr int GRID = 256;
constexpr int NCONS = 256;
constexpr int NTHREADS = 512;
#define PULL_LAG 8

constexpr int PK_UNIT = 15360;
constexpr int RAW_UNIT = 16384;
constexpr size_t RAW_OFF = 4u << 20;
constexpr size_t SLOT_BYTES = 8u << 20;
constexpr size_t WS_X16 = 0;
constexpr size_t WS_FLAGS = 2 * SLOT_BYTES;
constexpr size_t WS_INV = WS_FLAGS + (size_t)NR * 256 * 4;
constexpr size_t WS_WFLAGS = WS_INV + 32 * 128;
constexpr size_t WS_CNT = WS_WFLAGS + 16 * 64 * 4 * 4;
constexpr size_t WS_PROG = WS_CNT + (size_t)GRID * 128;
constexpr size_t WS_TMO = WS_PROG + (size_t)GRID * 128;
constexpr size_t WS_W16 = 17u << 20;
constexpr size_t WS_NEED = WS_W16 + (size_t)NCT * NKT * B_TILE;
static_assert(WS_TMO + 128 <= WS_W16, "workspace layout");
static_assert((size_t)256 * PK_UNIT <= RAW_OFF && RAW_OFF + (size_t)256 * RAW_UNIT <= SLOT_BYTES, "slot layout");
constexpr float XSCALE = 0.25f, OSCALE = 4.0f;
constexpr unsigned ZBITS = 0x40004000u;

constexpr u64 DEADLINE_TICKS = 30000000ull;

struct Params {
    const float* x;
    const float* w;
    float* out;
    char* ws[NR];
    int rank;
    int pad;
};

template <int V> struct IC { static constexpr int value = V; };

__device__ __forceinline__ f16x8 cvt8(f32x4 a, f32x4 b) {
    f16x2 p0 = __builtin_convertvector((f32x2){a[0], a[1]}, f16x2);
    f16x2 p1 = __builtin_convertvector((f32x2){a[2], a[3]}, f16x2);
    f16x2 p2 = __builtin_convertvector((f32x2){b[0], b[1]}, f16x2);
    f16x2 p3 = __builtin_convertvector((f32x2){b[2], b[3]}, f16x2);
    return (f16x8){p0[0], p0[1], p1[0], p1[1], p2[0], p2[1], p3[0], p3[1]};
}

__device__ __forceinline__ int pk_off16(int fr, int fq) { return fr * 64 + ((fq ^ ((0x78 >> (2 * (fr >> 2))) & 3)) << 4); }
__device__ __forceinline__ int pk_off8(int fr, int fq) { return 12288 + fr * 32 + ((fq ^ ((fr >> 3) << 1)) << 3); }
__device__ __forceinline__ int pk_off4(int fr, int fq) { return 14336 + fr * 16 + ((fq ^ ((fr >> 3) << 1)) << 2); }

#define GLDS_(sc, o) "global_load_lds_dwordx4 %0, %1 offset:" #o sc "\n\t"
template <int NP, bool SC>
__device__ __forceinline__ void glds(unsigned voff, const char* sbase, unsigned ldsaddr, bool last3 = false) {
    static_assert(NP == 1 || NP == 2 || NP == 4, "pieces");
    if constexpr (NP == 4) {
        if (last3) {
            if constexpr (SC) asm volatile("s_mov_b32 m0, %2\n\ts_nop 0\n\t" GLDS_(" sc0 sc1", 0) GLDS_(" sc0 sc1", 1024) GLDS_(" sc0 sc1", 2048) GLDS_(" sc0 sc1", 2048) :: "v"(voff), "s"(sbase), "s"(ldsaddr) : "memory");
            else asm volatile("s_mov_b32 m0, %2\n\ts_nop 0\n\t" GLDS_("", 0) GLDS_("", 1024) GLDS_("", 2048) GLDS_("", 2048) :: "v"(voff), "s"(sbase), "s"(ldsaddr) : "memory");
        } else {
            if constexpr (SC) asm volatile("s_mov_b32 m0, %2\n\ts_nop 0\n\t" GLDS_(" sc0 sc1", 0) GLDS_(" sc0 sc1", 1024) GLDS_(" sc0 sc1", 2048) GLDS_(" sc0 sc1", 3072) :: "v"(voff), "s"(sbase), "s"(ldsaddr) : "memory");
            else asm volatile("s_mov_b32 m0, %2\n\ts_nop 0\n\t" GLDS_("", 0) GLDS_("", 1024) GLDS_("", 2048) GLDS_("", 3072) :: "v"(voff), "s"(sbase), "s"(ldsaddr) : "memory");
        }
    } else if constexpr (NP == 2) {
        if (last3) asm volatile("s_mov_b32 m0, %2\n\ts_nop 0\n\t" GLDS_("", 0) GLDS_("", 0) :: "v"(voff), "s"(sbase), "s"(ldsaddr) : "memory");
        else if constexpr (SC) asm volatile("s_mov_b32 m0, %2\n\ts_nop 0\n\t" GLDS_(" sc0 sc1", 0) GLDS_(" sc0 sc1", 1024) :: "v"(voff), "s"(sbase), "s"(ldsaddr) : "memory");
        else asm volatile("s_mov_b32 m0, %2\n\ts_nop 0\n\t" GLDS_("", 0) GLDS_("", 1024) :: "v"(voff), "s"(sbase), "s"(ldsaddr) : "memory");
    } else {
        if constexpr (SC) asm volatile("s_mov_b32 m0, %2\n\ts_nop 0\n\t" GLDS_(" sc0 sc1", 0) :: "v"(voff), "s"(sbase), "s"(ldsaddr) : "memory");
        else asm volatile("s_mov_b32 m0, %2\n\ts_nop 0\n\t" GLDS_("", 0) :: "v"(voff), "s"(sbase), "s"(ldsaddr) : "memory");
    }
}
__device__ __forceinline__ void store_wt(void* d, f16x8 h) {
    asm volatile("global_store_dwordx4 %0, %1, off sc0 sc1\n\ts_nop 1" :: "v"(d), "v"(__builtin_bit_cast(u32x4, h)) : "memory");
}
__device__ __forceinline__ void store_u32_wt(GAS unsigned* d, unsigned v) {
    asm volatile("global_store_dword %0, %1, off sc0 sc1\n\ts_nop 1" :: "v"(d), "v"(v) : "memory");
}
template <int N> __device__ __forceinline__ void wait_v() {
    static_assert(N >= 0 && N < 64, "vmcnt");
    asm volatile("s_waitcnt vmcnt(%0)" :: "i"(N) : "memory");
}

#define WAIT_V(n) asm volatile("s_waitcnt vmcnt(" #n ")" ::: "memory")
#define WAIT_L(n) asm volatile("s_waitcnt lgkmcnt(" #n ")" ::: "memory")
#define BAR() do { asm volatile("" ::: "memory"); __builtin_amdgcn_s_barrier(); asm volatile("" ::: "memory"); } while (0)
#define SCHED() __builtin_amdgcn_sched_barrier(0)

#define STAMP(i) do {} while (0)
#define BWAIT_DECL do {} while (0)
#define BWAIT_A() do {} while (0)
#define BWAIT_B() do {} while (0)
#define BWAIT_FLUSH(slot) do {} while (0)
#define LSTAMP_DECL do {} while (0)
#define LSTAMP(t) do {} while (0)
#define CSTAMP(i) do {} while (0)
#define LSTAMP_FLUSH() do {} while (0)

template <class P> __device__ __forceinline__ void spin_flags(GAS u64* fp, GAS u64* fp2, GAS unsigned* tmo, unsigned call, P ok) {
    const u64 t0 = __builtin_amdgcn_s_memrealtime();
    for (;;) {
        const u64 f = __hip_atomic_load(fp, __ATOMIC_RELAXED, __HIP_MEMORY_SCOPE_SYSTEM);
        const u64 g = __hip_atomic_load(fp2, __ATOMIC_RELAXED, __HIP_MEMORY_SCOPE_SYSTEM);
        if (__all(ok(f) && ok(g))) break;
        if (__hip_atomic_load(tmo, __ATOMIC_RELAXED, __HIP_MEMORY_SCOPE_AGENT) == call ||
            __builtin_amdgcn_s_memrealtime() - t0 > DEADLINE_TICKS) {
            __hip_atomic_store(tmo, call, __ATOMIC_RELAXED, __HIP_MEMORY_SCOPE_AGENT);
            break;
        }
        __builtin_amdgcn_s_sleep(1);
    }
    WAIT_V(0);
}

template <class P> __device__ __forceinline__ bool spin_u64(GAS u64* fp, GAS unsigned* tmo, unsigned call, u64& f, P ok) {
    const u64 t0 = __builtin_amdgcn_s_memrealtime();
    for (;;) {
        f = __hip_atomic_load(fp, __ATOMIC_RELAXED, __HIP_MEMORY_SCOPE_SYSTEM);
        if (__all(ok(f))) return false;
        if (__hip_atomic_load(tmo, __ATOMIC_RELAXED, __HIP_MEMORY_SCOPE_AGENT) == call ||
            __builtin_amdgcn_s_memrealtime() - t0 > DEADLINE_TICKS) {
            __hip_atomic_store(tmo, call, __ATOMIC_RELAXED, __HIP_MEMORY_SCOPE_AGENT);
            return true;
        }
        __builtin_amdgcn_s_sleep(1);
    }
}

__global__ void __launch_bounds__(NTHREADS, 2) gemm_pull_kernel(Params p) {
    extern __shared__ __attribute__((aligned(16))) char smem[];
    LAS char* lds = (LAS char*)smem;
    const unsigned lds0 = (unsigned)(size_t)lds;
    const int tid = threadIdx.x;
    const int wid = __builtin_amdgcn_readfirstlane(tid >> 6);
    const int lane = tid & 63;
    const int b = blockIdx.x;
    const int label = b & 7, rr = b >> 3;
    const bool puller = rr >= 2 * NCT;
    const int q = puller ? rr - 2 * NCT : rr / NCT;
    const int ct = puller ? 0 : rr - q * NCT;
    const unsigned src = (unsigned)(p.rank + label) & 7u;
    char* const myws = p.ws[p.rank];
    GAS unsigned* const cnt = (GAS unsigned*)(myws + WS_CNT + (size_t)b * 128);
    GAS unsigned* const tmo = (GAS unsigned*)(myws + WS_TMO);
    LAS unsigned* const misc = (LAS unsigned*)(lds + LDS_MISC);
    char* const w16 = myws + WS_W16 + (size_t)ct * NKT * B_TILE;
    GAS unsigned* const wf = (GAS unsigned*)(myws + WS_WFLAGS) + ct * 256;

    GAS unsigned* const invw = (GAS unsigned*)(myws + WS_INV + (size_t)((unsigned)__builtin_amdgcn_s_getreg((3 << 11) | 20) & 0xFu) * 256);

    STAMP(0);
    unsigned call;
    size_t slot;
    bool raw;
    {
        const int rec = tid >> 1, j = tid & 1, rg = rec >> 6, fr = (rec >> 2) & 15, fq = rec & 3;
        const int urow = rg * 32 + fr + j * 16;
        const float* g = p.x + (size_t)((b >> 7) * 256 + (b & 1) * 128 + urow) * KD + ((b >> 1) & 63) * BK + fq * 8;
        const f32x4 v0 = *(const f32x4*)g, v1 = *(const f32x4*)(g + 4), v2 = *(const f32x4*)(g + 32), v3 = *(const f32x4*)(g + 36);
        GAS unsigned* fdst = nullptr;
        if (wid == 0) fdst = (GAS unsigned*)(p.ws[lane & 7] + WS_FLAGS) + p.rank * 256 + b;
        call = __builtin_amdgcn_readfirstlane(__hip_atomic_load(cnt, __ATOMIC_RELAXED, __HIP_MEMORY_SCOPE_AGENT)) + 1u;
        unsigned won = 0;
        if (wid == 4) {
            if (lane == 0) won = __hip_atomic_fetch_max(invw, call, __ATOMIC_RELAXED, __HIP_MEMORY_SCOPE_AGENT) < call ? 1u : 0u;
            won = __builtin_amdgcn_readfirstlane(won);
        }
        slot = WS_X16 + (size_t)(call & 1u) * SLOT_BYTES;
        u32x4 c0 = __builtin_bit_cast(u32x4, cvt8(v0 * XSCALE, v1 * XSCALE)), c1 = __builtin_bit_cast(u32x4, cvt8(v2 * XSCALE, v3 * XSCALE));
        const unsigned any = (c0[0] | c0[1] | c0[2] | c0[3] | c1[0] | c1[1] | c1[2] | c1[3]) & ZBITS;
        const bool wraw = __any(any != 0u);
        if (lane == 0) misc[8 + wid] = wraw ? 1u : 0u;
        WAIT_L(0);
        BAR();
        const u32x4 r0 = *(const LAS u32x4*)(misc + 8), r1 = *(const LAS u32x4*)(misc + 12);
        raw = (r0[0] | r0[1] | r0[2] | r0[3] | r1[0] | r1[1] | r1[2] | r1[3]) != 0u;
        if (!raw) {
            const unsigned tl = c1[3];
            const unsigned cc = (unsigned)__builtin_amdgcn_update_dpp(0, (int)tl, 0xB1  , 0xf, 0xf, false);
            const unsigned cb = j ? tl : cc;
#pragma unroll
            for (int e = 0; e < 4; ++e) {
                const int da = j * 8 + e, db = j * 8 + 4 + e;
                const int ia = (da & 1) ? 8 + (da >> 1) : (da >> 1);
                const int ib = db == 12 ? 7 : db == 13 ? 15 : db == 14 ? 6 : (db & 1) ? 8 + (db >> 1) : (db >> 1);
                c0[e] |= ((cb >> ia) & 0x00010001u) << 14;
                if (e < 3 || !j) c1[e] |= ((cb >> ib) & 0x00010001u) << 14;
            }
            char* u = myws + slot + (size_t)b * PK_UNIT;
            const int o16 = rg * 1024 + pk_off16(fr, fq);
            store_wt(u + (j ? 8192 : 0) + o16, __builtin_bit_cast(f16x8, c0));
            if (!j) {
                store_wt(u + 4096 + o16, __builtin_bit_cast(f16x8, c1));
            } else {
                asm volatile("global_store_dwordx2 %0, %1, off sc0 sc1\n\ts_nop 1" :: "v"(u + rg * 512 + pk_off8(fr, fq)), "v"((u32x2){c1[0], c1[1]}) : "memory");
                asm volatile("global_store_dword %0, %1, off sc0 sc1\n\ts_nop 1" :: "v"(u + rg * 256 + pk_off4(fr, fq)), "v"(c1[2]) : "memory");
            }
        } else {
            char* u = myws + slot + RAW_OFF + (size_t)b * RAW_UNIT + urow * 128;
            const int sw = (urow >> 1) & 7;
            store_wt(u + ((fq ^ sw) << 4), __builtin_bit_cast(f16x8, c0));
            store_wt(u + (((4 + fq) ^ sw) << 4), __builtin_bit_cast(f16x8, c1));
        }
        BAR();
        if (won) asm volatile("buffer_inv sc0 sc1" ::: "memory");
        WAIT_V(0);
        STAMP(1);
        BAR();
        if (won && lane == 0) __hip_atomic_store(invw + 32, call, __ATOMIC_RELAXED, __HIP_MEMORY_SCOPE_AGENT);
        if (wid == 0 && lane < NR) __hip_atomic_store(fdst, (call << 1) | (raw ? 1u : 0u), __ATOMIC_RELAXED, __HIP_MEMORY_SCOPE_SYSTEM);
        BAR();
    }

    if (wid >= 4 && !puller) {
        const int lw = wid - 4, pt = tid - NCONS;
        const int kt0 = label * 2 + q;
        const int colbase = p.rank * NS + ct * BN;
        const int colmax = p.rank * NS + NS - 4;
        const int kg = lane & 7;
        const int c1 = pt >> 3, c2 = 32 + lw;
        const float* g1 = p.w + (size_t)(kt0 * BK + kg * 8) * NT + min(colbase + c1 * 4, colmax);
        const float* g2 = p.w + (size_t)(kt0 * BK + kg * 8) * NT + min(colbase + c2 * 4, colmax);
        f32x4 wa[8], xa[8];
        auto WL = [&](auto part_, int r) {
            constexpr int PART = decltype(part_)::value;
            const size_t o = (size_t)r * 16 * BK * NT;
#pragma unroll
            for (int e = 0; e < 8; ++e)
                if (PART < 0 || (PART == 0 && e < 6) || (PART == 1 && e >= 6))
                    asm volatile("global_load_dwordx4 %0, %1, off nt" : "=v"(wa[e]) : "v"(g1 + o + (size_t)e * NT) : "memory");
            const float* h = g2 + o;
            if (PART < 0 || PART == 1)
                asm volatile("s_mov_b64 exec, 0xff\n\t"
                             "global_load_dwordx4 %0, %3, off nt\n\tglobal_load_dwordx4 %1, %4, off nt\n\tglobal_load_dwordx4 %2, %5, off nt\n\t"
                             "s_mov_b64 exec, -1"
                             : "=&v"(xa[0]), "=&v"(xa[1]), "=&v"(xa[2]) : "v"(h), "v"(h + NT), "v"(h + 2 * NT) : "memory");
            if (PART < 0 || PART == 2)
                asm volatile("s_mov_b64 exec, 0xff\n\t"
                             "global_load_dwordx4 %0, %5, off nt\n\tglobal_load_dwordx4 %1, %6, off nt\n\tglobal_load_dwordx4 %2, %7, off nt\n\t"
                             "global_load_dwordx4 %3, %8, off nt\n\tglobal_load_dwordx4 %4, %9, off nt\n\t"
                             "s_mov_b64 exec, -1"
                             : "=&v"(xa[3]), "=&v"(xa[4]), "=&v"(xa[5]), "=&v"(xa[6]), "=&v"(xa[7])
                             : "v"(h + 3 * NT), "v"(h + 4 * NT), "v"(h + 5 * NT), "v"(h + 6 * NT), "v"(h + 7 * NT) : "memory");
        };
        auto W_USE = [&]() {
            asm volatile("" : "+v"(wa[0]), "+v"(wa[1]), "+v"(wa[2]), "+v"(wa[3]), "+v"(wa[4]), "+v"(wa[5]), "+v"(wa[6]), "+v"(wa[7]),
                              "+v"(xa[0]), "+v"(xa[1]), "+v"(xa[2]), "+v"(xa[3]), "+v"(xa[4]), "+v"(xa[5]), "+v"(xa[6]), "+v"(xa[7]) :: "memory");
        };
        auto WS = [&](int r) {
            char* B = w16 + (size_t)(kt0 + 16 * r) * B_TILE;
            u32x4 h2[4];
            char* d2[4];
#pragma unroll
            for (int i = 0; i < 4; ++i) {
                const int row = 4 * c1 + i, row2 = 4 * c2 + i;
                store_wt(B + row * 128 + ((kg ^ ((row >> 1) & 7)) << 4),
                         cvt8((f32x4){wa[0][i], wa[1][i], wa[2][i], wa[3][i]}, (f32x4){wa[4][i], wa[5][i], wa[6][i], wa[7][i]}));
                h2[i] = __builtin_bit_cast(u32x4, cvt8((f32x4){xa[0][i], xa[1][i], xa[2][i], xa[3][i]}, (f32x4){xa[4][i], xa[5][i], xa[6][i], xa[7][i]}));
                d2[i] = B + row2 * 128 + ((kg ^ ((row2 >> 1) & 7)) << 4);
            }
            asm volatile("s_mov_b64 exec, 0xff\n\t"
                         "global_store_dwordx4 %0, %4, off sc0 sc1\n\tglobal_store_dwordx4 %1, %5, off sc0 sc1\n\t"
                         "global_store_dwordx4 %2, %6, off sc0 sc1\n\tglobal_store_dwordx4 %3, %7, off sc0 sc1\n\t"
                         "s_nop 1\n\ts_mov_b64 exec, -1"
                         :: "v"(d2[0]), "v"(d2[1]), "v"(d2[2]), "v"(d2[3]), "v"(h2[0]), "v"(h2[1]), "v"(h2[2]), "v"(h2[3]) : "memory");
        };
        WL(IC<-1>{}, 0);
        WAIT_V(0);
        W_USE();
        WS(0);
        WAIT_V(0);
        if (lane == 0) store_u32_wt(wf + kt0 * 4 + lw, call);
        STAMP(7);
        BAR();
        if (misc[0] != 0u) return;

        const u64 rawm0 = ((u64)__builtin_amdgcn_readfirstlane(misc[5]) << 32) | (unsigned)__builtin_amdgcn_readfirstlane(misc[4]);
        const u64 rawm1 = ((u64)__builtin_amdgcn_readfirstlane(misc[7]) << 32) | (unsigned)__builtin_amdgcn_readfirstlane(misc[6]);
        const char* upk = p.ws[src] + slot + (size_t)q * 128 * PK_UNIT;
        const char* urw = p.ws[src] + slot + RAW_OFF + (size_t)q * 128 * RAW_UNIT;
        const unsigned voff = (unsigned)lane * 16u + (unsigned)lw * 4096u;
        const unsigned la0 = lds0 + LDS_A + lw * 4096, lb0 = lds0 + LDS_B + lw * 4096;
        GAS unsigned* const prog = (GAS unsigned*)(myws + WS_PROG + (size_t)b * 128);
        const unsigned pbase = call * 64u;
        auto tile = [&](auto nb_, int T, int buf, auto&& mid) {
            constexpr int NB = decltype(nb_)::value;
#pragma unroll
            for (int h = 0; h < 2; ++h) {
                const bool rw = ((h ? rawm1 : rawm0) >> T) & 1u;
                const char* ga = rw ? urw + (size_t)(T * 2 + h) * RAW_UNIT : upk + (size_t)(T * 2 + h) * PK_UNIT;
                glds<4, false>(voff, ga, la0 + buf * A_TILE + h * 16384, !rw && lw == 3);
            }
            mid();
            const char* gb = w16 + (size_t)T * B_TILE;
            glds<4, true>(voff, gb, lb0 + buf * B_TILE);
            if constexpr (NB == 6) glds<2, true>(voff + 16384u, gb, lb0 + buf * B_TILE + 16384);
        };
        auto loop = [&](auto nb_, auto xp_) {
            constexpr int NB = decltype(nb_)::value;
            constexpr int N = NB + 8;
            constexpr int XP = decltype(xp_)::value;
            int t = 0, buf = 2;
            u32x4 sf;
            BWAIT_DECL;
            auto wcheck_issue = [&](int T) {
                const char* fp4 = (const char*)(wf + T * 4);
                asm volatile("s_load_dwordx4 %0, %1, 0x0 glc" : "=s"(sf) : "s"(fp4));
            };
            auto wcheck = [&](int T) {
                WAIT_L(0);
                asm volatile("" : "+s"(sf));
                if (min(min(sf[0], sf[1]), min(sf[2], sf[3])) < call) {
                    GAS u64* f2 = (GAS u64*)(wf + T * 4 + (lane & 1) * 2);
                    spin_flags(f2, f2, tmo, call, [&](u64 v) { return (unsigned)v >= call && (unsigned)(v >> 32) >= call; });
                }
            };
            auto none = [&]() {};
            auto step = [&](auto x_, auto xp_, auto&& extra, auto&& mid) {
                constexpr int X = decltype(x_)::value, XPREV = decltype(xp_)::value;
                BWAIT_A();
                if (lw == 0 && t + 3 < NKT) wcheck(t + 3);
                tile(nb_, t + 2, buf, mid);
                if (XP) store_u32_wt(prog, pbase + (unsigned)t - 1u);
                extra();
                if (lw == 0 && t + 4 < NKT) wcheck_issue(t + 4);
                wait_v<XP + XPREV + N + XP + X>();
                BWAIT_B();
                BAR();
                ++t; buf = (buf == 2) ? 0 : buf + 1;
            };
            tile(nb_, 0, 0, none); tile(nb_, 1, 1, none); tile(nb_, 2, 2, none);
            wait_v<2 * N>();
            BAR();
            if (lw == 0) {
                GAS u64* const f0 = (GAS u64*)(wf + 3 * 4 + (lane & 1) * 2);
                spin_flags(f0, f0, tmo, call, [&](u64 v) { return (unsigned)v >= call && (unsigned)(v >> 32) >= call; });
                wcheck_issue(4);
            }
            wait_v<N>();
            BAR();
            t = 1; buf = 0;
            step(IC<0>{}, IC<-XP>{}, none, none);
#pragma unroll 1
            for (int r = 1; r < 4; ++r) {
                const int base = r == 1 ? 2 : 16 * r - 17 + lw;
                while (t < base) step(IC<0>{}, IC<0>{}, none, none);
                step(IC<6>{}, IC<0>{}, none, [&]() { WL(IC<0>{}, r); });
                step(IC<5>{}, IC<0>{}, none, [&]() { WL(IC<1>{}, r); });
                step(IC<5>{}, IC<0>{}, none, [&]() { WL(IC<2>{}, r); });
                step(IC<0>{}, IC<0>{}, none, none);
                W_USE();
                step(IC<8>{}, IC<0>{}, [&]() { WS(r); }, none);
                step(IC<0>{}, IC<8>{}, none, none);
                step(IC<0>{}, IC<0>{}, none, none);
                step(IC<1>{}, IC<0>{}, [&]() { if (lane == 0) store_u32_wt(wf + (kt0 + 16 * r) * 4 + lw, call); }, none);
                step(IC<0>{}, IC<1>{}, none, none);
            }
            while (t < NKT - 2) step(IC<0>{}, IC<0>{}, none, none);
            WAIT_V(0);
            BAR();
            BWAIT_FLUSH(104 + wid);
        };
        if (lw == 0) loop(IC<6>{}, IC<0>{}); else if (lw == 3) loop(IC<4>{}, IC<1>{}); else loop(IC<4>{}, IC<0>{});
        return;
    }

    if (wid == 0) {
        STAMP(100);
        GAS u64* fp = (GAS u64*)(myws + WS_FLAGS + ((size_t)src * 256 + q * 128) * 4) + lane;
        u64 f;
        unsigned bad = spin_u64(fp, tmo, call, f, [&](u64 v) { return ((unsigned)v >> 1) >= call && (unsigned)(v >> 33) >= call; }) ? 1u : 0u;
        {
            const u64 m0 = __ballot((unsigned)f & 1u), m1 = __ballot((unsigned)(f >> 32) & 1u);
            if (lane == 0) { misc[4] = (unsigned)m0; misc[5] = (unsigned)(m0 >> 32); misc[6] = (unsigned)m1; misc[7] = (unsigned)(m1 >> 32); }
        }
        u64 g;
        bad |= spin_u64((GAS u64*)(invw + 32), tmo, call, g, [&](u64 v) { return (unsigned)v >= call; }) ? 1u : 0u;
        STAMP(101);
        if (!puller) {
            GAS u64* fw = (GAS u64*)(wf + min(lane >> 1, 2) * 4 + (lane & 1) * 2);
            spin_flags(fw, fw, tmo, call, [&](u64 v) { return (unsigned)v >= call && (unsigned)(v >> 32) >= call; });
            bad = __hip_atomic_load(tmo, __ATOMIC_RELAXED, __HIP_MEMORY_SCOPE_AGENT) == call ? 1u : 0u;
        }
        WAIT_V(0);
        STAMP(2);
        if (lane == 0) misc[0] = bad;
        WAIT_L(0);
        STAMP(3);
    }
    BAR();
    const bool bad = misc[0] != 0u;
    const u64 rawm0 = ((u64)__builtin_amdgcn_readfirstlane(misc[5]) << 32) | (unsigned)__builtin_amdgcn_readfirstlane(misc[4]);
    const u64 rawm1 = ((u64)__builtin_amdgcn_readfirstlane(misc[7]) << 32) | (unsigned)__builtin_amdgcn_readfirstlane(misc[6]);

    if (puller) {
        if (!bad) {
            const char* upk = p.ws[src] + slot + (size_t)q * 128 * PK_UNIT;
            const char* urw = p.ws[src] + slot + RAW_OFF + (size_t)q * 128 * RAW_UNIT;
            const unsigned l = lds0 + LDS_A + wid * 2048;
            const unsigned* prog = (const unsigned*)(myws + WS_PROG + (size_t)(q * NCT * 8 + label) * 128);
            const unsigned pbase = call * 64u;
            const unsigned vo0 = (unsigned)lane * 16u + (unsigned)wid * 2048u;
            unsigned seen = 0;
            bool throttle = true;
            LSTAMP_DECL;
            for (int t = 0; t < NKT; ++t) {
                if (t >= PULL_LAG && wid == 0 && throttle) {
                    const unsigned need = pbase + (unsigned)(t - PULL_LAG);
                    WAIT_L(0);
                    asm volatile("" : "+s"(seen));
                    if ((int)(seen - need) < 0) {
                        const u64 t0 = __builtin_amdgcn_s_memrealtime();
                        do {
                            seen = __builtin_amdgcn_readfirstlane(__hip_atomic_load((GAS unsigned*)prog, __ATOMIC_RELAXED, __HIP_MEMORY_SCOPE_SYSTEM));
                            if (__hip_atomic_load(tmo, __ATOMIC_RELAXED, __HIP_MEMORY_SCOPE_AGENT) == call ||
                                __builtin_amdgcn_s_memrealtime() - t0 > DEADLINE_TICKS) { throttle = false; break; }
                            __builtin_amdgcn_s_sleep(2);
                        } while ((int)(seen - need) < 0);
                    }
                }
                if (t >= PULL_LAG) BAR();
#pragma unroll
                for (int h = 0; h < 2; ++h) {
                    const bool rw = ((h ? rawm1 : rawm0) >> t) & 1u;
                    const char* g = rw ? urw + (size_t)(t * 2 + h) * RAW_UNIT : upk + (size_t)(t * 2 + h) * PK_UNIT;
                    glds<2, false>(vo0, g, l, !rw && wid == 7);
                }
                if (t + 1 >= PULL_LAG && wid == 0 && throttle) asm volatile("s_load_dword %0, %1, 0x0 glc" : "=s"(seen) : "s"(prog));
                WAIT_V(4);
                BAR();
                LSTAMP(t);
            }
            WAIT_V(0);
            WAIT_L(0);
            LSTAMP_FLUSH();
        }
        STAMP(4);
    } else {
        const int fr = lane & 15, fq = lane >> 4;
        const int nvalid = (NS - ct * BN) >> 4;
        f32x4 acc[4][9];
#pragma unroll
        for (int i = 0; i < 4; ++i)
#pragma unroll
            for (int j = 0; j < 9; ++j) acc[i][j] = (f32x4){0.f, 0.f, 0.f, 0.f};

        if (!bad) {
            const int sw = (fr >> 1) & 7;
            const int half = wid >> 1;
            const u64 rawm = half ? rawm1 : rawm0;
            const int aoff0 = LDS_A + half * 16384 + ((wid & 1) * 64 + fr) * 128 + ((fq ^ sw) << 4), aoff1 = aoff0 ^ 64;
            const int rg0 = (wid & 1) * 2;
            const int p16 = LDS_A + half * 16384 + rg0 * 1024 + pk_off16(fr, fq);
            const int p8 = LDS_A + half * 16384 + rg0 * 512 + pk_off8(fr, fq);
            const int p4 = LDS_A + half * 16384 + rg0 * 256 + pk_off4(fr, fq);
            const int boff0 = LDS_B + fr * 128 + ((fq ^ sw) << 4), boff1 = boff0 ^ 64;
            f16x8 a0[4], a1[4], b0[3], b1[3];
            u32x4 d0[2], d1[2], d2[2];
            u32x2 d3[2];
            unsigned d4[2];
#define LDA(dst, aoff, ab) do { _Pragma("unroll") for (int i_ = 0; i_ < 4; ++i_) dst[i_] = *(const LAS f16x8*)(lds + (ab) + (aoff) + i_ * 2048); } while (0)
#define LDB(dst, boff, bb, g) do { _Pragma("unroll") for (int j_ = 0; j_ < 3; ++j_) dst[j_] = *(const LAS f16x8*)(lds + (bb) + (boff) + ((g) * 3 + j_) * 2048); } while (0)
#define LDP(ab) do { _Pragma("unroll") for (int r_ = 0; r_ < 2; ++r_) { \
        d0[r_] = *(const LAS u32x4*)(lds + (ab) + p16 + r_ * 1024); d2[r_] = *(const LAS u32x4*)(lds + (ab) + p16 + r_ * 1024 + 8192); } \
        _Pragma("unroll") for (int r_ = 0; r_ < 2; ++r_) { d1[r_] = *(const LAS u32x4*)(lds + (ab) + p16 + r_ * 1024 + 4096); \
        d3[r_] = *(const LAS u32x2*)(lds + (ab) + p8 + r_ * 512); d4[r_] = *(const LAS unsigned*)(lds + (ab) + p4 + r_ * 256); } } while (0)
#define UNPK0() do { _Pragma("unroll") for (int r_ = 0; r_ < 2; ++r_) { \
        a0[2 * r_] = __builtin_bit_cast(f16x8, d0[r_] & ~ZBITS); a0[2 * r_ + 1] = __builtin_bit_cast(f16x8, d2[r_] & ~ZBITS); } } while (0)
#define PAIR(x, y) __builtin_amdgcn_perm((y), (x), 0x07030501u)
#define ANDOR(a, m, c) __builtin_amdgcn_bitop3_b32((a), (m), (c), 0xEA)
#define UNPK1(r_) do { \
        unsigned g_ = (PAIR(d0[r_][0], d0[r_][1]) >> 6) & 0x01010101u; \
        g_ = ANDOR(PAIR(d0[r_][2], d0[r_][3]) >> 5, 0x02020202u, g_); \
        g_ = ANDOR(PAIR(d2[r_][0], d2[r_][1]) >> 2, 0x10101010u, g_); \
        g_ = ANDOR(PAIR(d2[r_][2], d2[r_][3]) >> 1, 0x20202020u, g_); \
        g_ = ANDOR(PAIR(d1[r_][0], d1[r_][1]) >> 4, 0x04040404u, g_); \
        g_ = ANDOR(PAIR(d1[r_][2], d1[r_][3]) >> 3, 0x08080808u, g_); \
        g_ = ANDOR(PAIR(d3[r_][0], d3[r_][1]) << 1, 0x80808080u, g_); \
        g_ = ANDOR(d4[r_] >> 8, 0x00400040u, g_); \
        a1[2 * (r_)] = __builtin_bit_cast(f16x8, d1[r_] & ~ZBITS); \
        a1[2 * (r_) + 1] = __builtin_bit_cast(f16x8, (u32x4){d3[r_][0] & ~ZBITS, d3[r_][1] & ~ZBITS, d4[r_] & ~ZBITS, g_}); } while (0)
#define MMA(g, bf, af) do { _Pragma("unroll") for (int j_ = 0; j_ < 3; ++j_) _Pragma("unroll") for (int i_ = 0; i_ < 4; ++i_) \
        acc[i_][(g) * 3 + j_] = __builtin_amdgcn_mfma_f32_16x16x32_f16(bf[j_], af[i_], acc[i_][(g) * 3 + j_], 0, 0, 0); } while (0)
            BAR();
            LSTAMP_DECL;
            CSTAMP(0);
            int buf = 0;
            if (rawm == 0 && nvalid >= 9) {
                unsigned gq = 0, tq = 0;
                u32x4 x0[4], x1[4];
#define M1(i, j, g, bf, xf) acc[i][(g) * 3 + (j)] = __builtin_amdgcn_mfma_f32_16x16x32_f16(bf[j], __builtin_bit_cast(f16x8, xf[i]), acc[i][(g) * 3 + (j)], 0, 0, 0)
#define RB(dst, j, boff, bb, g) dst[j] = *(const LAS f16x8*)(lds + (bb) + (boff) + ((g) * 3 + (j)) * 2048)
#define CL(dst, src) dst = (src) & ~ZBITS
                auto gop = [&](auto r_, auto n_) {
                    constexpr int R = decltype(r_)::value, N = decltype(n_)::value;
                    if constexpr (N < 18) {
                        constexpr int K = N / 3, S = N % 3;
                        constexpr int P = K == 0 ? 0 : K == 1 ? 1 : K == 2 ? 4 : K == 3 ? 5 : K == 4 ? 2 : 3;
                        if constexpr (S == 0) {
                            const u32x4& w = P < 2 ? d0[R] : P < 4 ? d1[R] : d2[R];
                            tq = PAIR(w[(P & 1) * 2], w[(P & 1) * 2 + 1]);
                        } else if constexpr (S == 1) { tq >>= 6 - P; }
                        else if constexpr (K == 0) { gq = tq & 0x01010101u; }
                        else { gq = ANDOR(tq, 0x01010101u << P, gq); }
                    }
                    else if constexpr (N == 18) { tq = PAIR(d3[R][0], d3[R][1]); }
                    else if constexpr (N == 19) { tq <<= 1; }
                    else if constexpr (N == 20) { gq = ANDOR(tq, 0x80808080u, gq); }
                    else if constexpr (N == 21) { tq = d4[R] >> 8; }
                    else { x1[2 * R + 1][3] = ANDOR(tq, 0x00400040u, gq); }
                };
#define G(r, n) gop(IC<r>{}, IC<n>{})
#define RD16(dst, off) dst = *(const LAS u32x4*)(lds + (off))
#define RD8(dst, off) dst = *(const LAS u32x2*)(lds + (off))
#define RD4(dst, off) dst = *(const LAS unsigned*)(lds + (off))
#define WL_(n) __builtin_amdgcn_s_waitcnt(0xC07F | ((n) << 8))
                RD16(d0[0], p16); RD16(d2[0], p16 + 8192);
                RB(b0, 0, boff0, 0, 0); RB(b0, 1, boff0, 0, 0); RB(b0, 2, boff0, 0, 0);
                RD16(d0[1], p16 + 1024); RD16(d2[1], p16 + 1024 + 8192);
#pragma unroll
                for (int e = 0; e < 4; ++e) { CL(x0[0][e], d0[0][e]); CL(x0[1][e], d2[0][e]); }
                G(0, 0); G(0, 1); G(0, 2); G(0, 3); G(0, 4); G(0, 5);
                f16x8 b2[3];
                BWAIT_DECL;
                auto iter = [&](auto buf_, auto last_, int t) {
                    constexpr int BUF = decltype(buf_)::value, BN_ = BUF == 2 ? 0 : BUF + 1;
                    constexpr bool LAST = decltype(last_)::value != 0;
                    constexpr int ab = BUF * A_TILE, bb = BUF * B_TILE, an = BN_ * A_TILE, bbn = BN_ * B_TILE;
                    SCHED();
                    M1(0, 2, 0, b0, x0); WL_(0); RD16(d1[0], ab + p16 + 4096); SCHED();
                    M1(0, 1, 0, b0, x0); CL(x0[2][0], d0[1][0]); CL(x0[2][1], d0[1][1]); SCHED();
                    M1(0, 0, 0, b0, x0); CL(x0[2][2], d0[1][2]); RB(b1, 0, boff0, bb, 1); SCHED();
                    M1(1, 2, 0, b0, x0); CL(x0[2][3], d0[1][3]); RB(b1, 1, boff0, bb, 1); SCHED();
                    M1(1, 1, 0, b0, x0); CL(x0[3][0], d2[1][0]); RB(b1, 2, boff0, bb, 1); SCHED();
                    M1(1, 0, 0, b0, x0); CL(x0[3][1], d2[1][1]); CL(x0[3][2], d2[1][2]); SCHED();
                    M1(2, 2, 0, b0, x0); CL(x0[3][3], d2[1][3]); RD8(d3[0], ab + p8); SCHED();
                    M1(2, 1, 0, b0, x0); G(0, 6); RD4(d4[0], ab + p4); SCHED();
                    M1(2, 0, 0, b0, x0); G(0, 7); G(0, 8); SCHED();
                    M1(3, 2, 0, b0, x0); G(0, 9); G(0, 10); SCHED();
                    M1(3, 1, 0, b0, x0); G(0, 11); G(0, 12); SCHED();
                    M1(3, 0, 0, b0, x0); G(0, 13); WL_(2); SCHED();
                    M1(0, 2, 1, b1, x0); RB(b0, 0, boff0, bb, 2); G(0, 14); SCHED();
                    M1(1, 2, 1, b1, x0); G(0, 15); G(0, 16); SCHED();
                    M1(2, 2, 1, b1, x0); RB(b0, 1, boff0, bb, 2); G(0, 17); SCHED();
                    M1(3, 2, 1, b1, x0); RB(b0, 2, boff0, bb, 2); WL_(3); SCHED();
                    M1(0, 1, 1, b1, x0); G(0, 18); G(0, 19); SCHED();
                    M1(1, 1, 1, b1, x0); G(0, 20); G(0, 21); SCHED();
                    M1(2, 1, 1, b1, x0); G(0, 22); RD16(d1[1], ab + p16 + 1024 + 4096); SCHED();
                    M1(3, 1, 1, b1, x0); G(1, 0); G(1, 1); SCHED();
                    M1(0, 0, 1, b1, x0); G(1, 2); G(1, 3); SCHED();
                    M1(1, 0, 1, b1, x0); G(1, 4); RD8(d3[1], ab + p8 + 512); SCHED();
                    M1(2, 0, 1, b1, x0); G(1, 5); RD4(d4[1], ab + p4 + 256); SCHED();
                    M1(3, 0, 1, b1, x0); G(1, 6); WL_(3); SCHED();
                    M1(0, 2, 2, b0, x0); RB(b1, 0, boff1, bb, 0); G(1, 7); SCHED();
                    M1(1, 2, 2, b0, x0); G(1, 8); G(1, 9); SCHED();
                    M1(2, 2, 2, b0, x0); RB(b1, 1, boff1, bb, 0); G(1, 10); SCHED();
                    M1(3, 2, 2, b0, x0); G(1, 11); CL(x1[0][0], d1[0][0]); SCHED();
                    M1(0, 1, 2, b0, x0); RB(b1, 2, boff1, bb, 0); CL(x1[0][1], d1[0][1]); SCHED();
                    M1(1, 1, 2, b0, x0); CL(x1[0][2], d1[0][2]); CL(x1[0][3], d1[0][3]); SCHED();
                    M1(2, 1, 2, b0, x0); CL(x1[1][0], d3[0][0]); CL(x1[1][1], d3[0][1]); SCHED();
                    M1(3, 1, 2, b0, x0); CL(x1[1][2], d4[0]); WL_(3); SCHED();
                    M1(0, 0, 2, b0, x0); G(1, 12); G(1, 13); SCHED();
                    M1(1, 0, 2, b0, x0); G(1, 14); G(1, 15); SCHED();
                    M1(2, 0, 2, b0, x0); G(1, 16); G(1, 17); SCHED();
                    M1(3, 0, 2, b0, x0); WL_(0); G(1, 18); SCHED();
                    M1(0, 2, 0, b1, x1); CL(x1[2][0], d1[1][0]); RB(b0, 0, boff1, bb, 1); SCHED();
                    M1(0, 1, 0, b1, x1); CL(x1[2][1], d1[1][1]); CL(x1[2][2], d1[1][2]); SCHED();
                    M1(0, 0, 0, b1, x1); CL(x1[2][3], d1[1][3]); RB(b0, 1, boff1, bb, 1); SCHED();
                    M1(1, 2, 0, b1, x1); G(1, 19); RB(b0, 2, boff1, bb, 1); SCHED();
                    M1(1, 1, 0, b1, x1); G(1, 20); G(1, 21); SCHED();
                    M1(1, 0, 0, b1, x1); G(1, 22); CL(x1[3][0], d3[1][0]); SCHED();
                    M1(2, 2, 0, b1, x1); CL(x1[3][1], d3[1][1]); CL(x1[3][2], d4[1]); SCHED();
                    M1(2, 1, 0, b1, x1); SCHED();
                    M1(2, 0, 0, b1, x1); WL_(0); SCHED();
                    M1(0, 2, 1, b0, x1); RB(b2, 0, boff1, bb, 2); SCHED();
                    M1(1, 2, 1, b0, x1); RB(b2, 1, boff1, bb, 2); SCHED();
                    M1(2, 2, 1, b0, x1); RB(b2, 2, boff1, bb, 2); SCHED();
                    M1(0, 1, 1, b0, x1); M1(1, 1, 1, b0, x1); M1(2, 1, 1, b0, x1);
                    M1(0, 0, 1, b0, x1); M1(1, 0, 1, b0, x1); M1(2, 0, 1, b0, x1);
                    M1(3, 2, 0, b1, x1); M1(3, 1, 0, b1, x1); M1(3, 0, 0, b1, x1);
                    M1(3, 2, 1, b0, x1); M1(3, 1, 1, b0, x1); M1(3, 0, 1, b0, x1);
                    SCHED();
                    WL_(0);
                    if (!LAST) {
                        LSTAMP(t);
                        BWAIT_A();
                        BAR();
                        BWAIT_B();
                        M1(0, 2, 2, b2, x1); RD16(d0[0], an + p16); RD16(d2[0], an + p16 + 8192); SCHED();
                        M1(1, 2, 2, b2, x1); RB(b0, 0, boff0, bbn, 0); RB(b0, 1, boff0, bbn, 0); SCHED();
                        M1(2, 2, 2, b2, x1); RB(b0, 2, boff0, bbn, 0); RD16(d0[1], an + p16 + 1024); SCHED();
                        M1(3, 2, 2, b2, x1); RD16(d2[1], an + p16 + 1024 + 8192); SCHED();
                        M1(0, 1, 2, b2, x1); SCHED();
                        M1(1, 1, 2, b2, x1); WL_(5); CL(x0[0][0], d0[0][0]); SCHED();
                        M1(2, 1, 2, b2, x1); CL(x0[0][1], d0[0][1]); CL(x0[0][2], d0[0][2]); SCHED();
                        M1(3, 1, 2, b2, x1); CL(x0[0][3], d0[0][3]); CL(x0[1][0], d2[0][0]); SCHED();
                        M1(0, 0, 2, b2, x1); CL(x0[1][1], d2[0][1]); CL(x0[1][2], d2[0][2]); SCHED();
                        M1(1, 0, 2, b2, x1); CL(x0[1][3], d2[0][3]); G(0, 0); SCHED();
                        M1(2, 0, 2, b2, x1); G(0, 1); G(0, 2); G(0, 3); SCHED();
                        M1(3, 0, 2, b2, x1); G(0, 4); G(0, 5); WL_(2); SCHED();
                    } else {
#pragma unroll
                        for (int j = 0; j < 3; ++j)
#pragma unroll
                            for (int i = 0; i < 4; ++i) acc[i][6 + j] = __builtin_amdgcn_mfma_f32_16x16x32_f16(b2[j], __builtin_bit_cast(f16x8, x1[i]), acc[i][6 + j], 0, 0, 0);
                    }
                };
                for (int t = 0; t < NKT - 1; t += 3) { iter(IC<0>{}, IC<0>{}, t); iter(IC<1>{}, IC<0>{}, t + 1); iter(IC<2>{}, IC<0>{}, t + 2); }
                iter(IC<0>{}, IC<1>{}, NKT - 1);
                BWAIT_FLUSH(104 + wid);
            } else {
                const bool full = nvalid > 3;
                for (int t = 0; t < NKT; ++t) {
                    const int ab = buf * A_TILE, bb = buf * B_TILE;
                    if ((rawm >> t) & 1u) {
                        LDA(a0, aoff0, ab);
                        LDA(a1, aoff1, ab);
                    } else {
                        LDP(ab);
                        UNPK0();
                        UNPK1(0);
                        UNPK1(1);
                    }
                    LDB(b0, boff0, bb, 0);
                    MMA(0, b0, a0);
                    LDB(b0, boff1, bb, 0);
                    MMA(0, b0, a1);
                    if (full) {
#pragma unroll
                        for (int g = 1; g < 3; ++g) {
                            LDB(b0, boff0, bb, g);
                            MMA(g, b0, a0);
                            LDB(b0, boff1, bb, g);
                            MMA(g, b0, a1);
                        }
                    }
                    if (t + 1 < NKT) {
                        WAIT_L(0);
                        LSTAMP(t);
                        BAR();
                    }
                    buf = (buf == 2) ? 0 : buf + 1;
                }
            }
            CSTAMP(1);
            LSTAMP_FLUSH();
        }
        STAMP(4);
        const float qnan = __builtin_nanf("");
        const bool badf = bad || __hip_atomic_load(tmo, __ATOMIC_RELAXED, __HIP_MEMORY_SCOPE_AGENT) == call;
        float* const o = p.out + (size_t)(src * MS + q * BM + wid * 64) * NS + ct * BN + fq * 4;
        const int odd = ct & 1;
        const int rowA = fr & 7, colA = (fr >> 3) * 16;
        auto relu = [&](f32x4 v) {
            v *= OSCALE;
            v[0] = v[0] < 0.f ? 0.f : v[0]; v[1] = v[1] < 0.f ? 0.f : v[1]; v[2] = v[2] < 0.f ? 0.f : v[2]; v[3] = v[3] < 0.f ? 0.f : v[3];
            if (badf) v = (f32x4){qnan, qnan, qnan, qnan};
            return v;
        };
        auto st = [&](float* d, f32x4 v) { asm volatile("global_store_dwordx4 %0, %1, off sc1\n\ts_nop 1" :: "v"(d), "v"(v) : "memory"); };
        auto epi = [&](auto odd_) {
            constexpr int ODD = decltype(odd_)::value;
#pragma unroll
            for (int i = 0; i < 4; ++i) {
                float* const oi = o + (size_t)i * 16 * NS;
                constexpr int js = ODD ? 0 : 8;
                if (js < nvalid) st(oi + (size_t)fr * NS + js * 16, relu(acc[i][js]));
#pragma unroll
                for (int k = 0; k < 4; ++k) {
                    const int j = ODD + 2 * k;
                    if (j + 1 < nvalid) {
                        const f32x4 X = relu(acc[i][j]), Y = relu(acc[i][j + 1]);
                        f32x4 A, B;
#pragma unroll
                        for (int e = 0; e < 4; ++e) {
                            const float xe = X[e], ye = Y[e];
                            A[e] = __int_as_float(__builtin_amdgcn_update_dpp(__float_as_int(xe), __float_as_int(ye), 0x128, 0xf, 0xc, false));
                            B[e] = __int_as_float(__builtin_amdgcn_update_dpp(__float_as_int(ye), __float_as_int(xe), 0x128, 0xf, 0x3, false));
                        }
                        st(oi + (size_t)rowA * NS + j * 16 + colA, A);
                        st(oi + (size_t)(rowA + 8) * NS + j * 16 + colA, B);
                    } else if (j < nvalid) {
                        st(oi + (size_t)fr * NS + j * 16, relu(acc[i][j]));
                    }
                }
            }
        };
        if (odd) epi(IC<1>{}); else epi(IC<0>{});
        STAMP(5);
    }
    if (tid == 0) __hip_atomic_store(cnt, call, __ATOMIC_RELAXED, __HIP_MEMORY_SCOPE_AGENT);
}

}

extern "C" void kernel_launch(int rank, int nranks, ncclComm_t comm,
                              hipStream_t stream,
                              void* const* d_in, const int* in_sizes, int n_in,
                              void* d_out, int out_size,
                              const int* in_dtypes, int out_dtype,
                              void* const* d_out_all,
                              void* const* d_ws_all, size_t ws_bytes) {
    (void)comm; (void)in_dtypes; (void)out_dtype; (void)d_out_all;
    if (nranks != NR || n_in < 2 || (unsigned)rank >= (unsigned)NR || ws_bytes < WS_NEED ||
        in_sizes[0] != MS * KD || in_sizes[1] != KD * NT || out_size != NR * MS * NS) {
        fprintf(stderr, "kernel_launch: unexpected shapes\n");
        return;
    }
    static bool attr_set[64] = {};
    int dev = 0;
    HIP_CHECK(hipGetDevice(&dev));
    if (dev >= 0 && dev < 64 && !attr_set[dev]) {
        HIP_CHECK(hipFuncSetAttribute((const void*)gemm_pull_kernel, hipFuncAttributeMaxDynamicSharedMemorySize, LDS_BYTES));
        attr_set[dev] = true;
    }
    Params p;
    p.x = (const float*)d_in[0];
    p.w = (const float*)d_in[1];
    p.out = (float*)d_out;
    for (int i = 0; i < NR; ++i) p.ws[i] = (char*)d_ws_all[i];
    p.rank = rank;
    p.pad = 0;
    gemm_pull_kernel<<<GRID, NTHREADS, LDS_BYTES, stream>>>(p);
    HIP_CHECK(hipGetLastError());
}
```
